```python
import numpy as np

kernel_name = "gpu_ag_gemm_m2048_k2048_n2048_bf16_silu"


ABSMAX_RTOL = 0.008

COMPUTE_DTYPE = "bfloat16"
EPILOGUE = "silu"
IN_QUANT = None
OUT_QUANT = None
IN_DTYPES = {"x": "bfloat16", "w_mat": "bfloat16"}
WORLD_SIZE = 8

def _to_bf16(x):
    u = np.ascontiguousarray(x, dtype=np.float32).view(np.uint32)
    return ((u + 0x7FFF + ((u >> 16) & 1)) & 0xFFFF0000).view(np.float32)

def setup_inputs():

    x = _to_bf16(np.random.randn(2048, 2048).astype(np.float32))
    w_mat = _to_bf16((np.random.randn(2048, 2048) / np.sqrt(2048)).astype(np.float32))
    return {"x": x, "w_mat": w_mat}

def _epilogue(y):
    return (y / (1.0 + np.exp(-np.clip(y, -60.0, 60.0)))).astype(np.float32)

def reference(x, w_mat):
    y = (x @ w_mat).astype(np.float32)
    return _epilogue(y)
```

```cpp
#include <hip/hip_runtime.h>
#include <rccl/rccl.h>
#include <cstdio>

#define HIP_CHECK(expr) do { hipError_t _e = (expr); if (_e != hipSuccess) { \
    fprintf(stderr, "HIP error %s at %s:%d\n", hipGetErrorString(_e), __FILE__, __LINE__); \
    return; } } while (0)

namespace {

constexpr int NR = 8;
constexpr int KD = 2048;
constexpr int MS = 256;
constexpr int NS = 256;
constexpr int NWG = 256;
constexpr int NTHR = 512;
constexpr unsigned SHARD_BYTES = MS * KD * 2;
constexpr size_t WS_XCOPY = 0;
constexpr size_t WS_CTRL = 16ull << 20;
constexpr size_t CTRL_COUNTER = 0;
constexpr size_t CTRL_TMO = 128;
constexpr size_t CTRL_XFLAG = 4096;
constexpr size_t WS_NEED = WS_CTRL + CTRL_XFLAG + NR * NWG * 4;
constexpr unsigned long long DEADLINE_TICKS = 20000000ull;

constexpr int LDS_W = 0;
constexpr int LDS_RED = KD * 64;
constexpr int LDS_MISC = LDS_RED + 8192;
constexpr int LDS_BYTES = LDS_MISC + 64;

typedef unsigned v4u __attribute__((ext_vector_type(4)));
typedef unsigned v2u __attribute__((ext_vector_type(2)));
typedef short v8s __attribute__((ext_vector_type(8)));
typedef short v4s __attribute__((ext_vector_type(4)));
typedef float v4f __attribute__((ext_vector_type(4)));
typedef __bf16 v8bf __attribute__((ext_vector_type(8)));
typedef __bf16 v2bf __attribute__((ext_vector_type(2)));
typedef float v2f __attribute__((ext_vector_type(2)));
#define GAS __attribute__((address_space(1)))
#define LAS __attribute__((address_space(3)))
typedef GAS unsigned gu32;

struct Params {
    const char* x;
    const char* w;
    char* out;
    char* ws[NR];
    int rank;
};

__device__ __forceinline__ __amdgpu_buffer_rsrc_t make_rsrc(const void* p, unsigned bytes) {
    const unsigned long long a = (unsigned long long)p;
    const unsigned lo = __builtin_amdgcn_readfirstlane((unsigned)a);
    const unsigned hi = __builtin_amdgcn_readfirstlane((unsigned)(a >> 32));
    void* q = (void*)(((unsigned long long)hi << 32) | lo);
    return __builtin_amdgcn_make_buffer_rsrc(q, 0, bytes, 0x00020000);
}

__device__ __forceinline__ float silu(float y) {
    const float c = fminf(fmaxf(y, -60.0f), 60.0f);
    return y * __builtin_amdgcn_rcpf(1.0f + __expf(-c));
}

__device__ __forceinline__ unsigned pack_bf16(float lo, float hi) {
    v2f f = {lo, hi};
    v2bf h = __builtin_convertvector(f, v2bf);
    return __builtin_bit_cast(unsigned, h);
}

#define WAIT_VM0() __builtin_amdgcn_s_waitcnt(0x0F70)
#define ORDER() do { asm volatile("" ::: "memory"); __builtin_amdgcn_sched_barrier(0); } while (0)
#define BARRIER() do { asm volatile("s_waitcnt lgkmcnt(0)" ::: "memory"); __builtin_amdgcn_s_barrier(); asm volatile("" ::: "memory"); } while (0)

__global__ void __launch_bounds__(NTHR, 2) ag_gemm_silu(Params p) {
    extern __shared__ __attribute__((aligned(16))) char lds[];
    const int tid = threadIdx.x, lane = tid & 63;
    const int wave = __builtin_amdgcn_readfirstlane(tid >> 6);
    const int b = blockIdx.x;
    const int rank = p.rank;
    const int src = b & 7, j = b >> 3, rg = j >> 3, cg = j & 7;
    char* wsp[NR];
#pragma unroll
    for (int i = 0; i < NR; ++i) { wsp[i] = p.ws[i]; asm volatile("" : "+s"(wsp[i])); }
    char* const myws = p.ws[rank];
    gu32* const counter = (gu32*)(myws + WS_CTRL + CTRL_COUNTER);
    gu32* const tmo = (gu32*)(myws + WS_CTRL + CTRL_TMO);

    const unsigned ticket = __hip_atomic_load(counter, __ATOMIC_RELAXED, __HIP_MEMORY_SCOPE_AGENT);

    const char* wsrc = p.w + (size_t)(lane >> 2) * (NS * 2) + cg * 64 + (((lane & 3) ^ (((lane >> 5) & 1) << 1)) * 16);
    auto load_w = [&]() {
#pragma unroll
        for (int i = 0; i < 16; ++i) {
            const int blk = wave * 16 + i;
            __builtin_amdgcn_global_load_lds((const GAS unsigned*)(wsrc + (size_t)blk * 16 * (NS * 2)),
                                             (LAS unsigned*)(lds + LDS_W + blk * 1024), 16, 0, 0);
        }
    };

    unsigned call;
    if (wave < 4) {
        const int rt = b >> 4, ks = (b & 15) * 4 + wave;
        const v4u v = *(const GAS v4u*)(p.x + (size_t)(rt * 16 + (lane & 15)) * (KD * 2) + ks * 64 + (lane >> 4) * 16);
        ORDER();
        call = (unsigned)__builtin_amdgcn_readfirstlane(ticket) / NWG + 1;
        const __amdgpu_buffer_rsrc_t rc = make_rsrc(myws + WS_XCOPY + (size_t)(call & 1) * SHARD_BYTES, SHARD_BYTES);
        __builtin_amdgcn_raw_buffer_store_b128(v, rc, b * 4096 + wave * 1024 + lane * 16, 0, 17);
        ORDER();
        load_w();
        ORDER();
        asm volatile("s_waitcnt vmcnt(16)" ::: "memory");
    } else {
        load_w();
        ORDER();
        if (wave == 5) {
            asm volatile("buffer_inv sc0 sc1" ::: "memory");
        }
        call = (unsigned)__builtin_amdgcn_readfirstlane(ticket) / NWG + 1;
    }
    BARRIER();
    if (wave == 4) {
        if (lane < NR && lane != rank) {
            char* base = wsp[0];
#pragma unroll
            for (int i = 1; i < NR; ++i) base = (lane == i) ? wsp[i] : base;
            __hip_atomic_store((gu32*)(base + WS_CTRL + CTRL_XFLAG + (size_t)(rank * NWG + b) * 4), call,
                               __ATOMIC_RELAXED, __HIP_MEMORY_SCOPE_SYSTEM);
        }
        if (lane == 0) __hip_atomic_fetch_add(counter, 1u, __ATOMIC_RELAXED, __HIP_MEMORY_SCOPE_AGENT);
        unsigned bad = 0;
        if (src != rank) {
            gu32* f = (gu32*)(myws + WS_CTRL + CTRL_XFLAG + (size_t)(src * NWG + rg * 64 + lane) * 4);
            const unsigned long long t0 = __builtin_amdgcn_s_memrealtime();
            while (true) {
                const unsigned v = __hip_atomic_load(f, __ATOMIC_RELAXED, __HIP_MEMORY_SCOPE_SYSTEM);
                if (__all(v >= call)) break;
                const unsigned t = __hip_atomic_load(tmo, __ATOMIC_RELAXED, __HIP_MEMORY_SCOPE_AGENT);
                if (__builtin_amdgcn_readfirstlane(t) == call || __builtin_amdgcn_s_memrealtime() - t0 > DEADLINE_TICKS) {
                    if (lane == 0) __hip_atomic_store(tmo, call, __ATOMIC_RELAXED, __HIP_MEMORY_SCOPE_AGENT);
                    break;
                }
                __builtin_amdgcn_s_sleep(1);
            }
            const unsigned t = __hip_atomic_load(tmo, __ATOMIC_RELAXED, __HIP_MEMORY_SCOPE_AGENT);
            bad = (__builtin_amdgcn_readfirstlane(t) == call) ? 1u : 0u;
        }
        if (lane == 0) *(LAS unsigned*)(lds + LDS_MISC) = bad;
    }
    WAIT_VM0();
    BARRIER();
    const unsigned bad = __builtin_amdgcn_readfirstlane(*(LAS unsigned*)(lds + LDS_MISC));

    const int rtl = wave & 3, kh = wave >> 2;
    const int rt = rg * 4 + rtl;
    v4f acc0 = {0.f, 0.f, 0.f, 0.f}, acc1 = {0.f, 0.f, 0.f, 0.f};
    if (!bad) {
        const bool local = (src == rank);
        const char* abase = local ? p.x : (p.ws[src] + WS_XCOPY + (size_t)(call & 1) * SHARD_BYTES);
        const __amdgpu_buffer_rsrc_t ra = make_rsrc(abase, SHARD_BYTES);
        const unsigned voff = rt * 65536 + (local ? ((lane & 15) * (KD * 2) + (lane >> 4) * 16) : lane * 16);
        const unsigned kstride = local ? 64u : 1024u;
        v4u a[32];
#pragma unroll
        for (int i = 0; i < 32; ++i)
            a[i] = __builtin_amdgcn_raw_buffer_load_b128(ra, voff, (2 * i + kh) * kstride, 0);
        ORDER();
        const int g = lane >> 4, q = (lane >> 2) & 3, pp = lane & 3;
        const unsigned wl = LDS_W + kh * 2048 + (8 * g + q) * 64 + pp * 8;
        const LAS char* w0p = (const LAS char*)lds + wl + ((g & 1) * 32);
        const LAS char* w1p = (const LAS char*)lds + wl + (((g & 1) ^ 1) * 32);
#pragma unroll
        for (int i = 0; i < 32; ++i) {
            const v4s l0 = __builtin_amdgcn_ds_read_tr16_b64_v4i16((LAS v4s*)(w0p + i * 4096));
            const v4s h0 = __builtin_amdgcn_ds_read_tr16_b64_v4i16((LAS v4s*)(w0p + i * 4096 + 256));
            const v4s l1 = __builtin_amdgcn_ds_read_tr16_b64_v4i16((LAS v4s*)(w1p + i * 4096));
            const v4s h1 = __builtin_amdgcn_ds_read_tr16_b64_v4i16((LAS v4s*)(w1p + i * 4096 + 256));
            const v8s wf0 = __builtin_shufflevector(l0, h0, 0, 1, 2, 3, 4, 5, 6, 7);
            const v8s wf1 = __builtin_shufflevector(l1, h1, 0, 1, 2, 3, 4, 5, 6, 7);
            const v8bf xa = __builtin_bit_cast(v8bf, a[i]);
            acc0 = __builtin_amdgcn_mfma_f32_16x16x32_bf16(__builtin_bit_cast(v8bf, wf0), xa, acc0, 0, 0, 0);
            acc1 = __builtin_amdgcn_mfma_f32_16x16x32_bf16(__builtin_bit_cast(v8bf, wf1), xa, acc1, 0, 0, 0);
        }
    }
    LAS v4f* red = (LAS v4f*)(lds + LDS_RED + rtl * 2048) + lane;
    if (kh == 1) { red[0] = acc0; red[64] = acc1; }
    BARRIER();
    if (kh == 0) {
        acc0 += red[0]; acc1 += red[64];
        v2u o0, o1;
        if (!bad) {
            o0.x = pack_bf16(silu(acc0[0]), silu(acc0[1])); o0.y = pack_bf16(silu(acc0[2]), silu(acc0[3]));
            o1.x = pack_bf16(silu(acc1[0]), silu(acc1[1])); o1.y = pack_bf16(silu(acc1[2]), silu(acc1[3]));
        } else {
            o0.x = o0.y = o1.x = o1.y = 0x7FC07FC0u;
        }
        const int row = src * MS + rt * 16 + (lane & 15);
        char* op = p.out + (size_t)row * (NS * 2) + cg * 64 + (lane >> 4) * 8;
        *(GAS v2u*)op = o0;
        *(GAS v2u*)(op + 32) = o1;
    }
}

}

extern "C" void kernel_launch(int rank, int nranks, ncclComm_t comm,
                              hipStream_t stream,
                              void* const* d_in, const int* in_sizes, int n_in,
                              void* d_out, int out_size,
                              const int* in_dtypes, int out_dtype,
                              void* const* d_out_all,
                              void* const* d_ws_all, size_t ws_bytes) {
    if (nranks != NR || n_in != 2 || ws_bytes < WS_NEED || in_sizes[0] != MS * KD || in_sizes[1] != KD * NS ||
        out_size != NR * MS * NS || (unsigned)rank >= (unsigned)NR) {
        fprintf(stderr, "kernel_launch: unexpected problem geometry\n");
        return;
    }
    HIP_CHECK(hipFuncSetAttribute((const void*)ag_gemm_silu, hipFuncAttributeMaxDynamicSharedMemorySize, LDS_BYTES));
    Params p;
    p.x = (const char*)d_in[0];
    p.w = (const char*)d_in[1];
    p.out = (char*)d_out;
    for (int i = 0; i < NR; ++i) p.ws[i] = (char*)d_ws_all[i];
    p.rank = rank;
    hipLaunchKernelGGL(ag_gemm_silu, dim3(NWG), dim3(NTHR), LDS_BYTES, stream, p);
    HIP_CHECK(hipGetLastError());
}
```

```cpp
#include <hip/hip_runtime.h>
#include <rccl/rccl.h>
#include <cstddef>
#include <cstdio>

#define HIP_CHECK(expr) do { hipError_t _e = (expr); if (_e != hipSuccess) { \
    fprintf(stderr, "HIP error %s at %s:%d\n", hipGetErrorString(_e), __FILE__, __LINE__); \
    return; } } while (0)

namespace {

constexpr int NR = 8;
constexpr int KD = 2048;
constexpr int MS = 256;
constexpr int NS = 256;
constexpr int NWG = 256;
constexpr int NCW = 8;
constexpr int NTHR = 768;
#define ZNA 7
constexpr int NA = ZNA, NB = (16 - NA) / 3;
static_assert(NA + 3 * NB == 16, "tile split");
constexpr int NSLOT = NA + 6 * NB;
constexpr unsigned SHARD_BYTES = MS * KD * 2;
constexpr unsigned TILE_BYTES = KD * 32;
constexpr unsigned WST_BYTES = 2u << 20;
static_assert(NSLOT * TILE_BYTES <= WST_BYTES && NSLOT * 128 <= 16384, "staging");
constexpr size_t WS_XCOPY = 0;
constexpr size_t WS_WST = 2ull << 20;
constexpr size_t WS_CTRL = 16ull << 20;
constexpr size_t CTRL_COUNTER = 0;
constexpr size_t CTRL_TMO = 128;
constexpr size_t CTRL_XFLAG = 4096;
constexpr size_t CTRL_WFLAG = 16384;
constexpr size_t CTRL_DFLAG = 32768;
constexpr size_t WS_NEED = WS_CTRL + CTRL_DFLAG + NR * 1024 * 4;
#define STAMP(i) do { } while (0)
constexpr unsigned long long DEADLINE_TICKS = 20000000ull;

constexpr int LDS_W = 0;
constexpr int RED_ROW = 144;
constexpr int LDS_RED = KD * 64;
constexpr int LDS_SYNC = LDS_RED + 8 * 16 * RED_ROW;
constexpr int SY_XGO = 0, SY_WGO = 4, SY_BAD = 8, SY_WCNT = 12, SY_DMA = 16, SY_SENT = 20, SY_DONE = 24, SY_PAIR = 32, SY_XST = 64, SY_INV = 68, SY_PUSH = 72;
constexpr int LDS_BYTES = LDS_SYNC + 128;
#define XGT 550
#define XGATE (-1)
#define WIN_P 2
#define W_ORDER 2
#define WIN_B 1
constexpr unsigned long long LOCAL_TICKS = 500;
constexpr unsigned long long XGATE_TICKS = XGT;
#define ZWB 4
constexpr int WB = ZWB;

typedef unsigned v4u __attribute__((ext_vector_type(4)));
typedef unsigned v2u __attribute__((ext_vector_type(2)));
typedef short v8s __attribute__((ext_vector_type(8)));
typedef short v4s __attribute__((ext_vector_type(4)));
typedef float v4f __attribute__((ext_vector_type(4)));
typedef __bf16 v8bf __attribute__((ext_vector_type(8)));
typedef __bf16 v2bf __attribute__((ext_vector_type(2)));
typedef float v2f __attribute__((ext_vector_type(2)));
#define GAS __attribute__((address_space(1)))
#define LAS __attribute__((address_space(3)))
typedef GAS unsigned gu32;

struct Params {
    const char* x;
    const char* w;
    char* out[NR];
    char* ws[NR];
    int rank;
};

__device__ __forceinline__ __amdgpu_buffer_rsrc_t make_rsrc(const void* p, unsigned bytes) {
    const unsigned long long a = (unsigned long long)p;
    const unsigned lo = __builtin_amdgcn_readfirstlane((unsigned)a);
    const unsigned hi = __builtin_amdgcn_readfirstlane((unsigned)(a >> 32));
    void* q = (void*)(((unsigned long long)hi << 32) | lo);
    return __builtin_amdgcn_make_buffer_rsrc(q, 0, bytes, 0x00020000);
}
__device__ __forceinline__ float silu(float y) {
    const float c = fminf(fmaxf(y, -60.0f), 60.0f);
    return y * __builtin_amdgcn_rcpf(1.0f + __expf(-c));
}
__device__ __forceinline__ unsigned pack_bf16(float lo, float hi) {
    v2f f = {lo, hi};
    v2bf h = __builtin_convertvector(f, v2bf);
    return __builtin_bit_cast(unsigned, h);
}
__device__ __forceinline__ int group_off(int q, int m) {
    return q == 3 ? ((m << 1) | ((m ^ (m >> 1)) & 1)) : (((m >> q) << (q + 1)) | (m & ((1 << q) - 1)));
}
__device__ __forceinline__ void foreign_tile(int q, int tf, int& doff, int& slot) {
    if (tf < NA) { doff = 7; slot = tf; return; }
    const int u = tf - NA, bb = u / NB, r = u - bb * NB;
    const int s = (q == 3 || q == bb) ? 0 : 1;
    doff = (1 << bb) ^ (7 * s);
    slot = NA + (2 * bb + s) * NB + r;
}
__device__ __forceinline__ char* pick(unsigned long long tab, int i) {
    const unsigned lo = (unsigned)__builtin_amdgcn_ds_bpermute(i * 4, (int)(unsigned)tab);
    const unsigned hi = (unsigned)__builtin_amdgcn_ds_bpermute(i * 4, (int)(unsigned)(tab >> 32));
    return (char*)(((unsigned long long)hi << 32) | lo);
}

#define ORDER() do { asm volatile("" ::: "memory"); __builtin_amdgcn_sched_barrier(0); } while (0)
#define WAIT_VM0() do { __builtin_amdgcn_s_waitcnt(0x0F70); ORDER(); } while (0)
__device__ __forceinline__ void glds16(const void* gsrc, unsigned lds_dst) {
    unsigned keep;
    asm volatile("s_mov_b32 %0, m0\n\ts_mov_b32 m0, %2\n\ts_nop 0\n\tglobal_load_lds_dwordx4 %1, off\n\ts_mov_b32 m0, %0"
                 : "=&s"(keep) : "v"(gsrc), "s"(lds_dst) : "memory");
}
__device__ __forceinline__ void glds16_sys(const void* gsrc, unsigned lds_dst) {
    unsigned keep;
    asm volatile("s_mov_b32 %0, m0\n\ts_mov_b32 m0, %2\n\ts_nop 0\n\tglobal_load_lds_dwordx4 %1, off sc0 sc1\n\ts_mov_b32 m0, %0"
                 : "=&s"(keep) : "v"(gsrc), "s"(lds_dst) : "memory");
}
__device__ __forceinline__ unsigned lds_ld(unsigned addr) {
    unsigned v; asm volatile("ds_read_b32 %0, %1\n\ts_waitcnt lgkmcnt(0)" : "=v"(v) : "v"(addr) : "memory");
    return __builtin_amdgcn_readfirstlane(v);
}
__device__ __forceinline__ void lds_st(unsigned addr, unsigned v) {
    asm volatile("s_waitcnt lgkmcnt(0)\n\tds_write_b32 %0, %1" : : "v"(addr), "v"(v) : "memory");
}
__device__ __forceinline__ void lds_add(unsigned addr, unsigned v) {
    asm volatile("ds_add_u32 %0, %1" : : "v"(addr), "v"(v) : "memory");
}
__device__ __forceinline__ void lds_max(unsigned addr, unsigned v) {
    asm volatile("ds_max_u32 %0, %1" : : "v"(addr), "v"(v) : "memory");
}
__device__ __forceinline__ unsigned lds_add_rtn(unsigned addr, unsigned v) {
    unsigned r; asm volatile("ds_add_rtn_u32 %0, %1, %2\n\ts_waitcnt lgkmcnt(0)" : "=v"(r) : "v"(addr), "v"(v) : "memory");
    return r;
}
__device__ __forceinline__ void lds_wait_ge(unsigned addr, unsigned want) {
    for (unsigned n = 0; lds_ld(addr) < want && n < (1u << 24); ++n) __builtin_amdgcn_s_sleep(1);
}
__device__ __forceinline__ bool wait_flags(gu32* f, bool need, gu32* f2, bool need2, unsigned call, gu32* tmo, int lane) {
    const unsigned long long t0 = __builtin_amdgcn_s_memrealtime();
    while (true) {
        unsigned v = call, v2 = call;
        if (need) v = __hip_atomic_load(f, __ATOMIC_RELAXED, __HIP_MEMORY_SCOPE_SYSTEM);
        if (need2) v2 = __hip_atomic_load(f2, __ATOMIC_RELAXED, __HIP_MEMORY_SCOPE_SYSTEM);
        unsigned t = __hip_atomic_load(tmo, __ATOMIC_RELAXED, __HIP_MEMORY_SCOPE_AGENT);
        asm volatile("" : "+v"(t), "+v"(v), "+v"(v2));
        if (__all(v >= call && v2 >= call)) break;
        if (__builtin_amdgcn_readfirstlane(t) == call || __builtin_amdgcn_s_memrealtime() - t0 > DEADLINE_TICKS) {
            if (lane == 0) __hip_atomic_store(tmo, call, __ATOMIC_RELAXED, __HIP_MEMORY_SCOPE_AGENT);
            break;
        }
        __builtin_amdgcn_s_sleep(1);
    }
    const unsigned t = __hip_atomic_load(tmo, __ATOMIC_RELAXED, __HIP_MEMORY_SCOPE_AGENT);
    return __builtin_amdgcn_readfirstlane(t) == call;
}
__device__ __forceinline__ bool wait_flags(gu32* f, bool need, unsigned call, gu32* tmo, int lane) {
    return wait_flags(f, need, f, false, call, tmo, lane);
}

__global__ void __launch_bounds__(NTHR, 3) ag_gemm_silu(Params p) {
    extern __shared__ __attribute__((aligned(16))) char lds[];
    const int tid = threadIdx.x, lane = tid & 63;
    const int wave = __builtin_amdgcn_readfirstlane(tid >> 6);
    const int b = blockIdx.x;
    const int rank = p.rank;
    const int g8 = b & 7, j = b >> 3;
    const int q = g8 >> 1, mem = (g8 & 1) * 2 + (j >> 4), cp = j & 15;
    const int src = rank ^ group_off(q, mem);
    const bool foreign = cp >= 8;
    const int rot = (mem == 0) ? 4 * cp : 0;
    const unsigned long long tab =
        *(const GAS unsigned long long*)((const char*)__builtin_amdgcn_kernarg_segment_ptr() + offsetof(Params, out) + (lane & 15) * 8);
    char* const myws = p.ws[rank];
    gu32* const counter = (gu32*)(myws + WS_CTRL + CTRL_COUNTER);
    gu32* const tmo = (gu32*)(myws + WS_CTRL + CTRL_TMO);
    const unsigned lds0 = (unsigned)(size_t)(LAS char*)lds;
    const unsigned sy = lds0 + LDS_SYNC;

    const unsigned ticket = __hip_atomic_load(counter, __ATOMIC_RELAXED, __HIP_MEMORY_SCOPE_AGENT);

    int own0 = rank, own1 = rank, slot0 = 0, slot1 = 0, col0 = cp * 32, col1 = cp * 32 + 16;
    if (foreign) {
        int d0, d1;
        const int tf = (cp - 8) * 2;
        foreign_tile(q, tf, d0, slot0); foreign_tile(q, tf + 1, d1, slot1);
        own0 = rank ^ d0; own1 = rank ^ d1; col0 = tf * 16; col1 = tf * 16 + 16;
    }
    const int wc = (lane & 3) ^ (((lane >> 5) & 1) << 1);

    auto dma_share = [&](unsigned call) {
        lds_wait_ge(sy + SY_WGO, 1u);
        const char* st = myws + WS_WST + (size_t)(call & 1) * WST_BYTES;
        const char* wl = st + (size_t)((wc & 2) ? slot1 : slot0) * TILE_BYTES + (lane >> 2) * 32 + (wc & 1) * 16;
        unsigned mine = 0;
        while (true) {
            unsigned gidx = 0;
            if (lane == 0) gidx = lds_add_rtn(sy + SY_DMA, 1u);
            gidx = __builtin_amdgcn_readfirstlane(gidx);
            if (gidx >= 32) break;
#pragma unroll
            for (int i = 0; i < 4; ++i) {
                const unsigned blk = gidx * 4 + i;
                glds16(wl + (size_t)((blk + 2 * rot) & 127) * 512, lds0 + LDS_W + blk * 1024);
            }
            ++mine;
        }
        asm volatile("s_waitcnt vmcnt(0)" ::: "memory");
        if (lane == 0 && mine) lds_add(sy + SY_WCNT, mine);
    };

    if (wave >= NCW) {
        const int h = wave - NCW;
        const unsigned long long tstart = __builtin_amdgcn_s_memrealtime();
        if (h == 0) STAMP(0);
        const int fj = (j >> 4) * 8 + (j & 7);
        v4u v[2 * (NA > NB ? NA : NB) > 4 ? 2 * (NA > NB ? NA : NB) : 4];
        const int pc = (fj * 8 + g8) * 2 + (h >> 1);
        const int ji = q * 4 + mem, pr = cp - 8;
        const int pi = pr < 2 ? pr : pr - 2;
        const int u = (pi * 16 + ji) * 2 + (h >> 1);
        const bool psend = (pr == 2 || pr == 3);
        const int L = psend ? (h == 1 ? 6 : 7) : u % 6;
        const int wi = psend ? (pr - 2) * 16 + ji : u / 6;
        const int dd = (L == 6) ? 7 : ((1 << (L >> 1)) ^ (7 * (L & 1)));
        const int tile0 = (L == 6) ? 0 : NA + (L >> 1) * NB;
        const int slot0s = (L == 6) ? 0 : NA + L * NB;
        const int nt = (L == 6) ? NA : (L == 7 ? 0 : NB);
        constexpr int NTMAX = NA > NB ? NA : NB;
        if (foreign) {
            if (!(h & 1)) {
                const char* xs = p.x + (size_t)((pc >> 4) * 16 + (lane & 15)) * (KD * 2) + (pc & 15) * 256 + (lane >> 4) * 16;
#pragma unroll
                for (int f = 0; f < 4; ++f) v[f] = *(const GAS v4u*)(xs + f * 64);
            } else {
                const char* wsrc = p.w + (size_t)(wi * 32 + (lane >> 1)) * (NS * 2) + tile0 * 32 + (lane & 1) * 16;
#pragma unroll
                for (int t = 0; t < NTMAX; ++t)
                    if (t < nt) {
                        v[2 * t] = *(const GAS v4u*)(wsrc + t * 32);
                        v[2 * t + 1] = *(const GAS v4u*)(wsrc + (size_t)1024 * (NS * 2) + t * 32);
                    }
            }
        }
        if (h == 0) {
            uint4* z = (uint4*)(lds + LDS_SYNC);
            if (lane < 8) z[lane] = make_uint4(0u, 0u, 0u, 0u);
        }
        ORDER();
        asm volatile("s_waitcnt lgkmcnt(0)" ::: "memory");
        __builtin_amdgcn_s_barrier();
        ORDER();
        const unsigned call = (unsigned)__builtin_amdgcn_readfirstlane(ticket) / NWG + 1;
        if (foreign && !(h & 1)) {
            const __amdgpu_buffer_rsrc_t rc = make_rsrc(myws + WS_XCOPY + (size_t)(call & 1) * SHARD_BYTES, SHARD_BYTES);
#pragma unroll
            for (int f = 0; f < 4; ++f) __builtin_amdgcn_raw_buffer_store_b128(v[f], rc, pc * 4096 + f * 1024 + lane * 16, 0, 17);
            ORDER();
            if (lane == 0) lds_add(sy + SY_XST, 1u);
            WAIT_VM0();
            if (h == 0) STAMP(1);
            char* base = pick(tab, 8 + (rank ^ group_off(pc >> 6, lane & 3)));
            if (lane < 4)
                __hip_atomic_store((gu32*)(base + WS_CTRL + CTRL_XFLAG + (size_t)(rank * NWG + pc) * 4), call, __ATOMIC_RELAXED, __HIP_MEMORY_SCOPE_SYSTEM);
            ORDER();
            if (lane == 0) lds_add(sy + SY_SENT, 1u);
        } else if (foreign && nt) {
            char* dws = p.ws[rank ^ dd];
            const __amdgpu_buffer_rsrc_t rd = make_rsrc(dws + WS_WST + (size_t)(call & 1) * WST_BYTES + (size_t)slot0s * TILE_BYTES, (unsigned)nt * TILE_BYTES);
            gu32* fl = (gu32*)(dws + WS_CTRL + CTRL_WFLAG + (size_t)(slot0s * 32 + wi) * 4);
            WAIT_VM0();
            lds_wait_ge(sy + SY_SENT, 2u);
            if (h == 1) STAMP(11);
            const unsigned vo = wi * 1024 + lane * 16;
            if ((L != 6 && WIN_B == 1) || (L == 6 && WIN_P == 1)) {
#pragma unroll
                for (int t = 0; t < NTMAX; ++t)
                    if (t < nt) {
                        __builtin_amdgcn_raw_buffer_store_b128(v[2 * t], rd, vo, t * TILE_BYTES, 17);
                        ORDER();
                        if (t > 0) {
                            asm volatile("s_waitcnt vmcnt(1)" ::: "memory");
                            if (lane == 0) __hip_atomic_store(fl + (t - 1) * 32, call, __ATOMIC_RELAXED, __HIP_MEMORY_SCOPE_SYSTEM);
                            ORDER();
                        }
                        __builtin_amdgcn_raw_buffer_store_b128(v[2 * t + 1], rd, vo + 32768, t * TILE_BYTES, 17);
                        ORDER();
                        asm volatile("s_waitcnt vmcnt(1)" ::: "memory");
                    }
                asm volatile("s_waitcnt vmcnt(0)" ::: "memory");
                if (lane == 0) __hip_atomic_store(fl + (nt - 1) * 32, call, __ATOMIC_RELAXED, __HIP_MEMORY_SCOPE_SYSTEM);
            } else {
#pragma unroll
                for (int t = 0; t < NTMAX; ++t)
                    if (t < nt) {
                        if (t >= 2) {
                            asm volatile("s_waitcnt vmcnt(2)" ::: "memory");
                            if (lane == 0) __hip_atomic_store(fl + (t - 2) * 32, call, __ATOMIC_RELAXED, __HIP_MEMORY_SCOPE_SYSTEM);
                            ORDER();
                        }
                        __builtin_amdgcn_raw_buffer_store_b128(v[2 * t], rd, vo, t * TILE_BYTES, 17);
                        __builtin_amdgcn_raw_buffer_store_b128(v[2 * t + 1], rd, vo + 32768, t * TILE_BYTES, 17);
                        ORDER();
                    }
                asm volatile("s_waitcnt vmcnt(2)" ::: "memory");
                if (lane == 0) __hip_atomic_store(fl + (nt - 2) * 32, call, __ATOMIC_RELAXED, __HIP_MEMORY_SCOPE_SYSTEM);
                ORDER();
                asm volatile("s_waitcnt vmcnt(1)" ::: "memory");
                if (lane == 0) __hip_atomic_store(fl + (nt - 1) * 32, call, __ATOMIC_RELAXED, __HIP_MEMORY_SCOPE_SYSTEM);
            }
            if (h == 1) STAMP(2); else STAMP(12);
        }
        if (foreign && (h & 1) && lane == 0) lds_add(sy + SY_PUSH, 1u);
        if (h == 0) {
            if (lane == 0) __hip_atomic_fetch_add(counter, 1u, __ATOMIC_RELAXED, __HIP_MEMORY_SCOPE_AGENT);
            if (foreign) {
                const int sl = (lane & 32) ? slot1 : slot0;
                const bool bad = wait_flags((gu32*)(myws + WS_CTRL + CTRL_WFLAG + (size_t)(sl * 32 + (lane & 31)) * 4), true, call, tmo, lane);
                if (lane == 0) { if (bad) lds_st(sy + SY_BAD, 1u); lds_st(sy + SY_WGO, 1u); }
                STAMP(4);
            }
        } else if (h == 2) {
            bool bad = false;
            unsigned xgo = 1;
            if (src != rank) {
                const int d = src ^ rank, ld = (d == 1) ? 0 : (d == 6) ? 1 : (d == 2) ? 2 : (d == 5) ? 3 : (d == 4) ? 4 : 5;
                bad = wait_flags((gu32*)(myws + WS_CTRL + CTRL_XFLAG + (size_t)(src * NWG + q * 64 + lane) * 4), true,
                                 (gu32*)(myws + WS_CTRL + CTRL_WFLAG + (size_t)((NA + ld * NB + (XGATE >= 0 ? XGATE : 0)) * 32 + (lane & 31)) * 4), XGATE >= 0 && lane < 32,
                                 call, tmo, lane);
                while (__builtin_amdgcn_s_memrealtime() - tstart < XGATE_TICKS) __builtin_amdgcn_s_sleep(1);
            } else {
                gu32* f = (gu32*)(myws + WS_CTRL + CTRL_XFLAG + (size_t)(rank * NWG + q * 64 + lane) * 4);
                xgo = 2;
                do {
                    const unsigned v = __hip_atomic_load(f, __ATOMIC_RELAXED, __HIP_MEMORY_SCOPE_SYSTEM);
                    if (__all(v >= call)) { xgo = 1; break; }
                } while (__builtin_amdgcn_s_memrealtime() - tstart < LOCAL_TICKS);
            }
            if (lane == 0) { if (bad) lds_st(sy + SY_BAD, 1u); lds_st(sy + SY_XGO, xgo); }
            STAMP(3);
        }
        if (foreign) {
            dma_share(call);
        }
        if (h == 0 && (b & 7) != rank) {
            const int s = b & 7, i = b >> 3, jq = i >> 3, pr = (i & 1) * 4 + ((lane >> 3) & 3);
            int d0, d1, sl;
            foreign_tile(jq, 2 * pr, d0, sl); foreign_tile(jq, 2 * pr + 1, d1, sl);
            const bool need = lane < 32 && (d0 == (s ^ rank) || d1 == (s ^ rank));
            const bool bad = wait_flags((gu32*)(myws + WS_CTRL + CTRL_DFLAG + (size_t)(s * 1024 + i * 32 + (lane & 31)) * 4), need, call, tmo, lane);
            if (bad && lane == 0) *(GAS unsigned short*)(p.out[rank] + (size_t)rank * MS * (NS * 2)) = 0x7FC0;
            STAMP(5);
        }
        return;
    }

    const int rtl = wave >> 1, kh = wave & 1;
    if (!foreign) {
        const char* wl = p.w + (size_t)(lane >> 2) * (NS * 2) + cp * 64 + wc * 16;
#pragma unroll
        for (int i = 0; i < 16; ++i) {
            const int blk = wave * 16 + i;
            glds16(wl + (size_t)((blk + 2 * rot) & 127) * 16 * (NS * 2), lds0 + LDS_W + blk * 1024);
        }
    }
    ORDER();
    __builtin_amdgcn_s_barrier();
    ORDER();
    const unsigned call = (unsigned)__builtin_amdgcn_readfirstlane(ticket) / NWG + 1;
    if (!foreign) {
        asm volatile("s_waitcnt vmcnt(0)" ::: "memory");
        if (lane == 0) lds_add(sy + SY_WCNT, 4u);
    } else {
        lds_wait_ge(sy + SY_SENT, 2u);
        lds_wait_ge(sy + SY_PUSH, 2u);
        dma_share(call);
    }
    lds_wait_ge(sy + SY_XGO, 1u);
    bool bad = lds_ld(sy + SY_BAD) != 0;
    const bool local = lds_ld(sy + SY_XGO) == 2;
    const char* abase = local ? p.x : (p.ws[src] + WS_XCOPY + (size_t)(call & 1) * SHARD_BYTES);
    const __amdgpu_buffer_rsrc_t ra = make_rsrc(abase, bad ? 0u : SHARD_BYTES);
    const unsigned voff = (q * 4 + rtl) * 65536 + (local ? ((lane & 15) * (KD * 2) + (lane >> 4) * 16) : lane * 16);
    const unsigned kstride = local ? 64u : 1024u;
    v4u a[32];
    v4f acc0 = {0.f, 0.f, 0.f, 0.f}, acc1 = {0.f, 0.f, 0.f, 0.f};
    const unsigned done = sy + SY_DONE + kh * 4;
    auto issue = [&](int jb) {
#pragma unroll
        for (int i = 8 * jb; i < 8 * jb + 8; ++i)
            a[i] = __builtin_amdgcn_raw_buffer_load_b128(ra, voff, ((2 * i + kh + rot) & 63) * kstride, 0);
        ORDER();
    };
    const int g = lane >> 4, r4 = (lane >> 2) & 3, pp = lane & 3;
    const unsigned wlo = LDS_W + kh * 2048 + (8 * g + r4) * 64 + pp * 8;
    const LAS char* w0p = (const LAS char*)lds + wlo + ((g & 1) * 32);
    const LAS char* w1p = (const LAS char*)lds + wlo + (((g & 1) ^ 1) * 32);
    auto consume = [&](int jb) {
#pragma unroll
        for (int i = 8 * jb; i < 8 * jb + 8; ++i) {
            const v4s l0 = __builtin_amdgcn_ds_read_tr16_b64_v4i16((LAS v4s*)(w0p + i * 4096));
            const v4s h0 = __builtin_amdgcn_ds_read_tr16_b64_v4i16((LAS v4s*)(w0p + i * 4096 + 256));
            const v4s l1 = __builtin_amdgcn_ds_read_tr16_b64_v4i16((LAS v4s*)(w1p + i * 4096));
            const v4s h1 = __builtin_amdgcn_ds_read_tr16_b64_v4i16((LAS v4s*)(w1p + i * 4096 + 256));
            const v8s wf0 = __builtin_shufflevector(l0, h0, 0, 1, 2, 3, 4, 5, 6, 7);
            const v8s wf1 = __builtin_shufflevector(l1, h1, 0, 1, 2, 3, 4, 5, 6, 7);
            const v8bf xa = __builtin_bit_cast(v8bf, a[i]);
            acc0 = __builtin_amdgcn_mfma_f32_16x16x32_bf16(__builtin_bit_cast(v8bf, wf0), xa, acc0, 0, 0, 0);
            acc1 = __builtin_amdgcn_mfma_f32_16x16x32_bf16(__builtin_bit_cast(v8bf, wf1), xa, acc1, 0, 0, 0);
        }
    };
    if (foreign) {
#pragma unroll
        for (int jb = 0; jb < 4; ++jb) issue(jb);
        if (wave == 0) STAMP(6);
        lds_wait_ge(sy + SY_WCNT, 32u);
        bad = bad || lds_ld(sy + SY_BAD) != 0;
        if (wave == 0) STAMP(7);
#pragma unroll
        for (int jb = 0; jb < 4; ++jb) consume(jb);
    } else {
#pragma unroll
        for (int jb = 0; jb < WB; ++jb) {
            const int need = 32 * rtl + 8 * jb + 8 - 8 * WB;
            if (need > 0) lds_wait_ge(done, (unsigned)need);
            issue(jb);
        }
        if (wave == 0) STAMP(6);
        lds_wait_ge(sy + SY_WCNT, 32u);
        if (wave == 0) STAMP(7);
#pragma unroll
        for (int jb = 0; jb < 4; ++jb) {
            consume(jb);
            ORDER();
            if (lane == 0) lds_max(done, 32 * rtl + 8 * jb + 8);
            if (jb + WB < 4) issue(jb + WB);
        }
    }
    if (wave == 0) STAMP(8);
    if (wave == 7) STAMP(9);
    {
        char* red = lds + LDS_RED + rtl * (2 * 16 * RED_ROW);
        char* mine = red + kh * (16 * RED_ROW) + (lane & 15) * RED_ROW + (lane >> 4) * 16;
        *(v4f*)mine = acc0;
        *(v4f*)(mine + 64) = acc1;
        if (lane == 0) lds_st(sy + SY_PAIR + wave * 4, 1u);
        lds_wait_ge(sy + SY_PAIR + (wave ^ 1) * 4, 1u);
        const int row = kh * 8 + (lane >> 3), c4 = lane & 7;
        const v4f s0 = *(const v4f*)(red + row * RED_ROW + c4 * 16);
        const v4f s1 = *(const v4f*)(red + 16 * RED_ROW + row * RED_ROW + c4 * 16);
        const v4f y = s0 + s1;
        v2u o;
        if (!bad) { o.x = pack_bf16(silu(y[0]), silu(y[1])); o.y = pack_bf16(silu(y[2]), silu(y[3])); }
        else { o.x = o.y = 0x7FC07FC0u; }
        const int grow = src * MS + q * 64 + rtl * 16 + row;
        const bool t1 = c4 >= 4;
        char* ob = pick(tab, t1 ? own1 : own0);
        char* op = ob + (size_t)grow * (NS * 2) + (t1 ? col1 : col0) * 2 + (c4 & 3) * 8;
        if (!foreign) {
            *(GAS v2u*)op = o;
        } else {
            asm volatile("global_store_dwordx2 %0, %1, off sc0 sc1\n\ts_waitcnt vmcnt(0)" : : "v"(op), "v"(o) : "memory");
            char* base = pick(tab, 8 + (lane ? own1 : own0));
            if (lane < 2 && (lane == 0 || own1 != own0)) {
                __hip_atomic_store((gu32*)(base + WS_CTRL + CTRL_DFLAG + (size_t)(rank * 1024 + ((q * 4 + mem) * 8 + (cp - 8)) * 8 + wave) * 4), call,
                                   __ATOMIC_RELAXED, __HIP_MEMORY_SCOPE_SYSTEM);
            }
        }
    }
    if (wave == 7) STAMP(10);
}

}

extern "C" void kernel_launch(int rank, int nranks, ncclComm_t comm,
                              hipStream_t stream,
                              void* const* d_in, const int* in_sizes, int n_in,
                              void* d_out, int out_size,
                              const int* in_dtypes, int out_dtype,
                              void* const* d_out_all,
                              void* const* d_ws_all, size_t ws_bytes) {
    if (nranks != NR || n_in != 2 || ws_bytes < WS_NEED || in_sizes[0] != MS * KD || in_sizes[1] != KD * NS ||
        out_size != NR * MS * NS || (unsigned)rank >= (unsigned)NR) {
        fprintf(stderr, "kernel_launch: unexpected problem geometry\n");
        return;
    }
    HIP_CHECK(hipFuncSetAttribute((const void*)ag_gemm_silu, hipFuncAttributeMaxDynamicSharedMemorySize, LDS_BYTES));
    Params p;
    p.x = (const char*)d_in[0];
    p.w = (const char*)d_in[1];
    for (int i = 0; i < NR; ++i) { p.out[i] = (char*)d_out_all[i]; p.ws[i] = (char*)d_ws_all[i]; }
    p.out[rank] = (char*)d_out;
    p.rank = rank;
    hipLaunchKernelGGL(ag_gemm_silu, dim3(NWG), dim3(NTHR), LDS_BYTES, stream, p);
    HIP_CHECK(hipGetLastError());
}
```

```cpp
#include <hip/hip_runtime.h>
#include <rccl/rccl.h>
#include <cstddef>
#include <cstdio>

#define HIP_CHECK(expr) do { hipError_t _e = (expr); if (_e != hipSuccess) { \
    fprintf(stderr, "HIP error %s at %s:%d\n", hipGetErrorString(_e), __FILE__, __LINE__); \
    return; } } while (0)

namespace {

constexpr int NR = 8;
constexpr int KD = 2048;
constexpr int MS = 256;
constexpr int NS = 256;
constexpr int NWG = 256;
constexpr int NCW = 8;
constexpr int NTHR = 768;
#define ZNA 7
constexpr int NA = ZNA, NB = (16 - NA) / 3;
static_assert(NA + 3 * NB == 16, "tile split");
constexpr int NSLOT = NA + 6 * NB;
constexpr unsigned SHARD_BYTES = MS * KD * 2;
constexpr unsigned TILE_BYTES = KD * 32;
constexpr unsigned WST_BYTES = 2u << 20;
static_assert(NSLOT * TILE_BYTES <= WST_BYTES && NSLOT * 128 <= 16384, "staging");
constexpr size_t WS_XCOPY = 0;
constexpr size_t WS_WST = 2ull << 20;
constexpr size_t WS_CTRL = 16ull << 20;
constexpr size_t CTRL_COUNTER = 0;
constexpr size_t CTRL_TMO = 128;
constexpr size_t CTRL_XFLAG = 4096;
constexpr size_t CTRL_WFLAG = 16384;
constexpr size_t CTRL_DFLAG = 32768;
constexpr size_t WS_NEED = WS_CTRL + CTRL_DFLAG + NR * 1024 * 4;
#define STAMP(i) do { } while (0)
constexpr unsigned long long DEADLINE_TICKS = 20000000ull;

constexpr int LDS_W = 0;
constexpr int RED_ROW = 144;
constexpr int LDS_RED = KD * 64;
constexpr int LDS_SYNC = LDS_RED + 8 * 16 * RED_ROW;
constexpr int SY_XGO = 0, SY_WGO = 4, SY_BAD = 8, SY_WCNT = 12, SY_DMA = 16, SY_SENT = 20, SY_DONE = 24, SY_PAIR = 32, SY_XST = 64, SY_INV = 68, SY_PUSH = 72, SY_WR = 80;
constexpr int LDS_BYTES = LDS_SYNC + 128;
#define XGT 500
#define XGATE (-1)
#define WIN_P 1
#define W_ORDER 0
#define SPLIT_XCD 0
#define PROGRESSIVE 0
#define WIN_B 1
constexpr unsigned long long LOCAL_TICKS = 500;
constexpr unsigned long long XGATE_TICKS = XGT;
#define ZWB 4
constexpr int WB = ZWB;

typedef unsigned v4u __attribute__((ext_vector_type(4)));
typedef unsigned v2u __attribute__((ext_vector_type(2)));
typedef short v8s __attribute__((ext_vector_type(8)));
typedef short v4s __attribute__((ext_vector_type(4)));
typedef float v4f __attribute__((ext_vector_type(4)));
typedef __bf16 v8bf __attribute__((ext_vector_type(8)));
typedef __bf16 v2bf __attribute__((ext_vector_type(2)));
typedef float v2f __attribute__((ext_vector_type(2)));
#define GAS __attribute__((address_space(1)))
#define LAS __attribute__((address_space(3)))
typedef GAS unsigned gu32;

struct Params {
    const char* x;
    const char* w;
    char* out[NR];
    char* ws[NR];
    int rank;
};

__device__ __forceinline__ __amdgpu_buffer_rsrc_t make_rsrc(const void* p, unsigned bytes) {
    const unsigned long long a = (unsigned long long)p;
    const unsigned lo = __builtin_amdgcn_readfirstlane((unsigned)a);
    const unsigned hi = __builtin_amdgcn_readfirstlane((unsigned)(a >> 32));
    void* q = (void*)(((unsigned long long)hi << 32) | lo);
    return __builtin_amdgcn_make_buffer_rsrc(q, 0, bytes, 0x00020000);
}
__device__ __forceinline__ float silu(float y) {
    const float c = fminf(fmaxf(y, -60.0f), 60.0f);
    return y * __builtin_amdgcn_rcpf(1.0f + __expf(-c));
}
__device__ __forceinline__ unsigned pack_bf16(float lo, float hi) {
    v2f f = {lo, hi};
    v2bf h = __builtin_convertvector(f, v2bf);
    return __builtin_bit_cast(unsigned, h);
}
__device__ __forceinline__ int group_off(int q, int m) {
    return q == 3 ? ((m << 1) | ((m ^ (m >> 1)) & 1)) : (((m >> q) << (q + 1)) | (m & ((1 << q) - 1)));
}
__device__ __forceinline__ void foreign_tile(int q, int tf, int& doff, int& slot) {
    if (tf < NA) { doff = 7; slot = tf; return; }
    const int u = tf - NA, bb = u / NB, r = u - bb * NB;
    const int s = (q == 3 || q == bb) ? 0 : 1;
    doff = (1 << bb) ^ (7 * s);
    slot = NA + (2 * bb + s) * NB + r;
}
__device__ __forceinline__ char* pick(unsigned long long tab, int i) {
    const unsigned lo = (unsigned)__builtin_amdgcn_ds_bpermute(i * 4, (int)(unsigned)tab);
    const unsigned hi = (unsigned)__builtin_amdgcn_ds_bpermute(i * 4, (int)(unsigned)(tab >> 32));
    return (char*)(((unsigned long long)hi << 32) | lo);
}

#define ORDER() do { asm volatile("" ::: "memory"); __builtin_amdgcn_sched_barrier(0); } while (0)
#define WAIT_VM0() do { __builtin_amdgcn_s_waitcnt(0x0F70); ORDER(); } while (0)
__device__ __forceinline__ void glds16(const void* gsrc, unsigned lds_dst) {
    unsigned keep;
    asm volatile("s_mov_b32 %0, m0\n\ts_mov_b32 m0, %2\n\ts_nop 0\n\tglobal_load_lds_dwordx4 %1, off\n\ts_mov_b32 m0, %0"
                 : "=&s"(keep) : "v"(gsrc), "s"(lds_dst) : "memory");
}
__device__ __forceinline__ void glds16_sys(const void* gsrc, unsigned lds_dst) {
    unsigned keep;
    asm volatile("s_mov_b32 %0, m0\n\ts_mov_b32 m0, %2\n\ts_nop 0\n\tglobal_load_lds_dwordx4 %1, off sc0 sc1\n\ts_mov_b32 m0, %0"
                 : "=&s"(keep) : "v"(gsrc), "s"(lds_dst) : "memory");
}
__device__ __forceinline__ unsigned lds_ld(unsigned addr) {
    unsigned v; asm volatile("ds_read_b32 %0, %1\n\ts_waitcnt lgkmcnt(0)" : "=v"(v) : "v"(addr) : "memory");
    return __builtin_amdgcn_readfirstlane(v);
}
__device__ __forceinline__ void lds_st(unsigned addr, unsigned v) {
    asm volatile("s_waitcnt lgkmcnt(0)\n\tds_write_b32 %0, %1" : : "v"(addr), "v"(v) : "memory");
}
__device__ __forceinline__ void lds_add(unsigned addr, unsigned v) {
    asm volatile("ds_add_u32 %0, %1" : : "v"(addr), "v"(v) : "memory");
}
__device__ __forceinline__ void lds_max(unsigned addr, unsigned v) {
    asm volatile("ds_max_u32 %0, %1" : : "v"(addr), "v"(v) : "memory");
}
__device__ __forceinline__ unsigned lds_add_rtn(unsigned addr, unsigned v) {
    unsigned r; asm volatile("ds_add_rtn_u32 %0, %1, %2\n\ts_waitcnt lgkmcnt(0)" : "=v"(r) : "v"(addr), "v"(v) : "memory");
    return r;
}
__device__ __forceinline__ void lds_wait_ge(unsigned addr, unsigned want) {
    for (unsigned n = 0; lds_ld(addr) < want && n < (1u << 24); ++n) __builtin_amdgcn_s_sleep(1);
}
__device__ __forceinline__ bool wait_flags(gu32* f, bool need, gu32* f2, bool need2, unsigned call, gu32* tmo, int lane) {
    const unsigned long long t0 = __builtin_amdgcn_s_memrealtime();
    while (true) {
        unsigned v = call, v2 = call;
        if (need) v = __hip_atomic_load(f, __ATOMIC_RELAXED, __HIP_MEMORY_SCOPE_SYSTEM);
        if (need2) v2 = __hip_atomic_load(f2, __ATOMIC_RELAXED, __HIP_MEMORY_SCOPE_SYSTEM);
        unsigned t = __hip_atomic_load(tmo, __ATOMIC_RELAXED, __HIP_MEMORY_SCOPE_AGENT);
        asm volatile("" : "+v"(t), "+v"(v), "+v"(v2));
        if (__all(v >= call && v2 >= call)) break;
        if (__builtin_amdgcn_readfirstlane(t) == call || __builtin_amdgcn_s_memrealtime() - t0 > DEADLINE_TICKS) {
            if (lane == 0) __hip_atomic_store(tmo, call, __ATOMIC_RELAXED, __HIP_MEMORY_SCOPE_AGENT);
            break;
        }
        __builtin_amdgcn_s_sleep(1);
    }
    const unsigned t = __hip_atomic_load(tmo, __ATOMIC_RELAXED, __HIP_MEMORY_SCOPE_AGENT);
    return __builtin_amdgcn_readfirstlane(t) == call;
}
__device__ __forceinline__ bool wait_flags(gu32* f, bool need, unsigned call, gu32* tmo, int lane) {
    return wait_flags(f, need, f, false, call, tmo, lane);
}

__global__ void __launch_bounds__(NTHR, 3) ag_gemm_silu(Params p) {
    extern __shared__ __attribute__((aligned(16))) char lds[];
    const int tid = threadIdx.x, lane = tid & 63;
    const int wave = __builtin_amdgcn_readfirstlane(tid >> 6);
    const int b = blockIdx.x;
    const int rank = p.rank;
    const int g8 = b & 7, j = b >> 3;
    const int q = g8 >> 1, mem = (g8 & 1) * 2 + (j >> 4), cp = j & 15;
    const int src = rank ^ group_off(q, mem);
    const bool foreign = cp >= 8;
    const int rot = (mem == 0 && cp < 8) ? 4 * cp : 0;
    const unsigned long long tab =
        *(const GAS unsigned long long*)((const char*)__builtin_amdgcn_kernarg_segment_ptr() + offsetof(Params, out) + (lane & 15) * 8);
    char* const myws = p.ws[rank];
    gu32* const counter = (gu32*)(myws + WS_CTRL + CTRL_COUNTER);
    gu32* const tmo = (gu32*)(myws + WS_CTRL + CTRL_TMO);
    const unsigned lds0 = (unsigned)(size_t)(LAS char*)lds;
    const unsigned sy = lds0 + LDS_SYNC;

    const unsigned ticket = __hip_atomic_load(counter, __ATOMIC_RELAXED, __HIP_MEMORY_SCOPE_AGENT);

    int own0 = rank, own1 = rank, slot0 = 0, slot1 = 0, col0 = cp * 32, col1 = cp * 32 + 16;
    if (foreign) {
        int d0, d1;
        const int tf = (cp - 8) * 2;
        foreign_tile(q, tf, d0, slot0); foreign_tile(q, tf + 1, d1, slot1);
        own0 = rank ^ d0; own1 = rank ^ d1; col0 = tf * 16; col1 = tf * 16 + 16;
    }
    const int wc = (lane & 3) ^ (((lane >> 5) & 1) << 1);

    if (wave >= NCW) {
        const int h = wave - NCW;
        const unsigned long long tstart = __builtin_amdgcn_s_memrealtime();
        if (h == 0) STAMP(0);
        const int fj = (j >> 4) * 8 + (j & 7);
        v4u v[2 * (NA > NB ? NA : NB) > 4 ? 2 * (NA > NB ? NA : NB) : 4];
        const int pr = cp - 8;
        const int pi = pr < 2 ? pr : pr - 2;
        const bool psend = (pr == 2 || pr == 3);
        const bool isx = !(h & 1);
        const int pc = (fj * 8 + g8) * 2 + (h >> 1);
        const int ji = q * 4 + mem;
        const int u = (pi * 16 + ji) * 2 + (h >> 1);
        const int L = psend ? (h == 1 ? 6 : 7) : u % 6;
        const int wi = psend ? (pr - 2) * 16 + ji : u / 6;
        const int dd = (L == 6) ? 7 : ((1 << (L >> 1)) ^ (7 * (L & 1)));
        const int tile0 = (L == 6) ? 0 : NA + (L >> 1) * NB;
        const int slot0s = (L == 6) ? 0 : NA + L * NB;
        const int nt = (L == 6) ? NA : (L == 7 ? 0 : NB);
        constexpr int NTMAX = NA > NB ? NA : NB;
        if (foreign) {
            if (isx) {
                const char* xs = p.x + (size_t)((pc >> 4) * 16 + (lane & 15)) * (KD * 2) + (pc & 15) * 256 + (lane >> 4) * 16;
#pragma unroll
                for (int f = 0; f < 4; ++f) v[f] = *(const GAS v4u*)(xs + f * 64);
            } else {
                const int pl = lane ^ (((lane >> 4) & 1) << 3);
                const char* wsrc = p.w + (size_t)(wi * 32 + (pl >> 1)) * (NS * 2) + tile0 * 32 + (pl & 1) * 16;
#pragma unroll
                for (int t = 0; t < NTMAX; ++t)
                    if (t < nt) {
                        v[2 * t] = *(const GAS v4u*)(wsrc + t * 32);
                        v[2 * t + 1] = *(const GAS v4u*)(wsrc + (size_t)1024 * (NS * 2) + t * 32);
                    }
            }
        }
        if (h == 0) {
            uint4* z = (uint4*)(lds + LDS_SYNC);
            if (lane < 8) z[lane] = make_uint4(0u, 0u, 0u, 0u);
        }
        ORDER();
        asm volatile("s_waitcnt lgkmcnt(0)" ::: "memory");
        __builtin_amdgcn_s_barrier();
        ORDER();
        const unsigned call = (unsigned)__builtin_amdgcn_readfirstlane(ticket) / NWG + 1;
        if (foreign && isx) {
            const __amdgpu_buffer_rsrc_t rc = make_rsrc(myws + WS_XCOPY + (size_t)(call & 1) * SHARD_BYTES, SHARD_BYTES);
#pragma unroll
            for (int f = 0; f < 4; ++f) __builtin_amdgcn_raw_buffer_store_b128(v[f], rc, pc * 4096 + f * 1024 + lane * 16, 0, 17);
            ORDER();
            if (lane == 0) lds_add(sy + SY_XST, 1u);
            WAIT_VM0();
            if (h == 0 || h == 2) STAMP(1);
            char* base = pick(tab, 8 + (rank ^ group_off(pc >> 6, lane & 3)));
            if (lane < 4)
                __hip_atomic_store((gu32*)(base + WS_CTRL + CTRL_XFLAG + (size_t)(rank * NWG + pc) * 4), call, __ATOMIC_RELAXED, __HIP_MEMORY_SCOPE_SYSTEM);
            ORDER();
        } else if (foreign && nt) {
            char* dws = p.ws[rank ^ dd];
            const __amdgpu_buffer_rsrc_t rd = make_rsrc(dws + WS_WST + (size_t)(call & 1) * WST_BYTES + (size_t)slot0s * TILE_BYTES, (unsigned)nt * TILE_BYTES);
            gu32* fl = (gu32*)(dws + WS_CTRL + CTRL_WFLAG + (size_t)(slot0s * 32 + wi) * 4);
            WAIT_VM0();
            if (h == 1) STAMP(11);
            const unsigned vo = wi * 1024 + lane * 16;
            if ((L != 6 && WIN_B == 1) || (L == 6 && WIN_P == 1)) {
#pragma unroll
                for (int t = 0; t < NTMAX; ++t)
                    if (t < nt) {
                        __builtin_amdgcn_raw_buffer_store_b128(v[2 * t], rd, vo, t * TILE_BYTES, 17);
                        ORDER();
                        if (t > 0) {
                            asm volatile("s_waitcnt vmcnt(1)" ::: "memory");
                            if (lane == 0) __hip_atomic_store(fl + (t - 1) * 32, call, __ATOMIC_RELAXED, __HIP_MEMORY_SCOPE_SYSTEM);
                            ORDER();
                        }
                        __builtin_amdgcn_raw_buffer_store_b128(v[2 * t + 1], rd, vo + 32768, t * TILE_BYTES, 17);
                        ORDER();
                        asm volatile("s_waitcnt vmcnt(1)" ::: "memory");
                    }
                asm volatile("s_waitcnt vmcnt(0)" ::: "memory");
                if (lane == 0) __hip_atomic_store(fl + (nt - 1) * 32, call, __ATOMIC_RELAXED, __HIP_MEMORY_SCOPE_SYSTEM);
            } else {
#pragma unroll
                for (int t = 0; t < NTMAX; ++t)
                    if (t < nt) {
                        if (t >= 2) {
                            asm volatile("s_waitcnt vmcnt(2)" ::: "memory");
                            if (lane == 0) __hip_atomic_store(fl + (t - 2) * 32, call, __ATOMIC_RELAXED, __HIP_MEMORY_SCOPE_SYSTEM);
                            ORDER();
                        }
                        __builtin_amdgcn_raw_buffer_store_b128(v[2 * t], rd, vo, t * TILE_BYTES, 17);
                        __builtin_amdgcn_raw_buffer_store_b128(v[2 * t + 1], rd, vo + 32768, t * TILE_BYTES, 17);
                        ORDER();
                    }
                asm volatile("s_waitcnt vmcnt(2)" ::: "memory");
                if (lane == 0) __hip_atomic_store(fl + (nt - 2) * 32, call, __ATOMIC_RELAXED, __HIP_MEMORY_SCOPE_SYSTEM);
                ORDER();
                asm volatile("s_waitcnt vmcnt(1)" ::: "memory");
                if (lane == 0) __hip_atomic_store(fl + (nt - 1) * 32, call, __ATOMIC_RELAXED, __HIP_MEMORY_SCOPE_SYSTEM);
            }
            if (h == 1) STAMP(2); else STAMP(12);
        }
        if (foreign && lane == 0) lds_add(sy + SY_PUSH, 1u);
        if (h == 0) {
            if (lane == 0) __hip_atomic_fetch_add(counter, 1u, __ATOMIC_RELAXED, __HIP_MEMORY_SCOPE_AGENT);
            if (foreign) {
                const int sl = (lane & 32) ? slot1 : slot0;
                const bool bad = wait_flags((gu32*)(myws + WS_CTRL + CTRL_WFLAG + (size_t)(sl * 32 + (lane & 31)) * 4), true, call, tmo, lane);
                if (lane == 0) { if (bad) lds_st(sy + SY_BAD, 1u); lds_st(sy + SY_WGO, 1u); }
                STAMP(4);
            }
        } else if (h == 2) {
            bool bad = false;
            unsigned xgo = 1;
            if (src != rank) {
                const int d = src ^ rank, ld = (d == 1) ? 0 : (d == 6) ? 1 : (d == 2) ? 2 : (d == 5) ? 3 : (d == 4) ? 4 : 5;
                bad = wait_flags((gu32*)(myws + WS_CTRL + CTRL_XFLAG + (size_t)(src * NWG + q * 64 + lane) * 4), true,
                                 (gu32*)(myws + WS_CTRL + CTRL_WFLAG + (size_t)((NA + ld * NB + (XGATE >= 0 ? XGATE : 0)) * 32 + (lane & 31)) * 4), XGATE >= 0 && lane < 32,
                                 call, tmo, lane);
                while (__builtin_amdgcn_s_memrealtime() - tstart < XGATE_TICKS) __builtin_amdgcn_s_sleep(1);
            } else {
                gu32* f = (gu32*)(myws + WS_CTRL + CTRL_XFLAG + (size_t)(rank * NWG + q * 64 + lane) * 4);
                xgo = 2;
                do {
                    const unsigned v = __hip_atomic_load(f, __ATOMIC_RELAXED, __HIP_MEMORY_SCOPE_SYSTEM);
                    if (__all(v >= call)) { xgo = 1; break; }
                } while (__builtin_amdgcn_s_memrealtime() - tstart < LOCAL_TICKS);
            }
            if (lane == 0) { if (bad) lds_st(sy + SY_BAD, 1u); lds_st(sy + SY_XGO, xgo); }
            STAMP(3);
        }
        if (h == 0 && (b & 7) != rank) {
            const int s = b & 7, i = b >> 3, jq = i >> 3, pr = (i & 1) * 4 + ((lane >> 3) & 3);
            int d0, d1, sl;
            foreign_tile(jq, 2 * pr, d0, sl); foreign_tile(jq, 2 * pr + 1, d1, sl);
            const bool need = lane < 32 && (d0 == (s ^ rank) || d1 == (s ^ rank));
            const bool bad = wait_flags((gu32*)(myws + WS_CTRL + CTRL_DFLAG + (size_t)(s * 1024 + i * 32 + (lane & 31)) * 4), need, call, tmo, lane);
            if (bad && lane == 0) *(GAS unsigned short*)(p.out[rank] + (size_t)rank * MS * (NS * 2)) = 0x7FC0;
            STAMP(5);
        }
        return;
    }

    const int rtl = wave >> 1, kh = wave & 1;
    if (!foreign) {
        const char* wl = p.w + (size_t)(lane >> 2) * (NS * 2) + cp * 64 + wc * 16;
#pragma unroll
        for (int i = 0; i < 16; ++i) {
            const int blk = wave * 16 + i;
            glds16(wl + (size_t)((blk + 2 * rot) & 127) * 16 * (NS * 2), lds0 + LDS_W + blk * 1024);
        }
    }
    ORDER();
    __builtin_amdgcn_s_barrier();
    ORDER();
    const unsigned call = (unsigned)__builtin_amdgcn_readfirstlane(ticket) / NWG + 1;
    if (!foreign) {
        asm volatile("s_waitcnt vmcnt(0)" ::: "memory");
        if (lane == 0) lds_add(sy + SY_WCNT, 4u);
    } else {
        lds_wait_ge(sy + SY_PUSH, 4u);
        lds_wait_ge(sy + SY_WGO, 1u);
    }
    lds_wait_ge(sy + SY_XGO, 1u);
    bool bad = lds_ld(sy + SY_BAD) != 0;
    const bool local = lds_ld(sy + SY_XGO) == 2;
    const char* abase = local ? p.x : (p.ws[src] + WS_XCOPY + (size_t)(call & 1) * SHARD_BYTES);
    const __amdgpu_buffer_rsrc_t ra = make_rsrc(abase, bad ? 0u : SHARD_BYTES);
    const unsigned voff = (q * 4 + rtl) * 65536 + (local ? ((lane & 15) * (KD * 2) + (lane >> 4) * 16) : lane * 16);
    const unsigned kstride = local ? 64u : 1024u;
    v4u a[32];
    v4f acc0 = {0.f, 0.f, 0.f, 0.f}, acc1 = {0.f, 0.f, 0.f, 0.f};
    const unsigned done = sy + SY_DONE + kh * 4;
    auto issue = [&](int jb) {
#pragma unroll
        for (int i = 8 * jb; i < 8 * jb + 8; ++i)
            a[i] = __builtin_amdgcn_raw_buffer_load_b128(ra, voff, ((2 * i + kh + rot) & 63) * kstride, 0);
        ORDER();
    };
    const int g = lane >> 4, r4 = (lane >> 2) & 3, pp = lane & 3;
    const unsigned wlo = LDS_W + kh * 2048 + (8 * g + r4) * 64 + pp * 8;
    const LAS char* w0p = (const LAS char*)lds + wlo + ((g & 1) * 32);
    const LAS char* w1p = (const LAS char*)lds + wlo + (((g & 1) ^ 1) * 32);
    auto consume = [&](int jb) {
#pragma unroll
        for (int i = 8 * jb; i < 8 * jb + 8; ++i) {
            const v4s l0 = __builtin_amdgcn_ds_read_tr16_b64_v4i16((LAS v4s*)(w0p + i * 4096));
            const v4s h0 = __builtin_amdgcn_ds_read_tr16_b64_v4i16((LAS v4s*)(w0p + i * 4096 + 256));
            const v4s l1 = __builtin_amdgcn_ds_read_tr16_b64_v4i16((LAS v4s*)(w1p + i * 4096));
            const v4s h1 = __builtin_amdgcn_ds_read_tr16_b64_v4i16((LAS v4s*)(w1p + i * 4096 + 256));
            const v8s wf0 = __builtin_shufflevector(l0, h0, 0, 1, 2, 3, 4, 5, 6, 7);
            const v8s wf1 = __builtin_shufflevector(l1, h1, 0, 1, 2, 3, 4, 5, 6, 7);
            const v8bf xa = __builtin_bit_cast(v8bf, a[i]);
            acc0 = __builtin_amdgcn_mfma_f32_16x16x32_bf16(__builtin_bit_cast(v8bf, wf0), xa, acc0, 0, 0, 0);
            acc1 = __builtin_amdgcn_mfma_f32_16x16x32_bf16(__builtin_bit_cast(v8bf, wf1), xa, acc1, 0, 0, 0);
        }
    };
    const unsigned flo = LDS_W + kh * 1024 + (8 * g + r4) * 32 + pp * 8;
    const LAS char* f0l = (const LAS char*)lds + flo + ((g & 1) * 128);
    const LAS char* f0h = (const LAS char*)lds + flo + (((g & 1) ^ 1) * 128);
    const LAS char* f1l = f0l + TILE_BYTES;
    const LAS char* f1h = f0h + TILE_BYTES;
    auto consume_f = [&](int jb) {
#pragma unroll
        for (int i = 8 * jb; i < 8 * jb + 8; ++i) {
            const v4s l0 = __builtin_amdgcn_ds_read_tr16_b64_v4i16((LAS v4s*)(f0l + i * 2048));
            const v4s h0 = __builtin_amdgcn_ds_read_tr16_b64_v4i16((LAS v4s*)(f0h + i * 2048));
            const v4s l1 = __builtin_amdgcn_ds_read_tr16_b64_v4i16((LAS v4s*)(f1l + i * 2048));
            const v4s h1 = __builtin_amdgcn_ds_read_tr16_b64_v4i16((LAS v4s*)(f1h + i * 2048));
            const v8s wf0 = __builtin_shufflevector(l0, h0, 0, 1, 2, 3, 4, 5, 6, 7);
            const v8s wf1 = __builtin_shufflevector(l1, h1, 0, 1, 2, 3, 4, 5, 6, 7);
            const v8bf xa = __builtin_bit_cast(v8bf, a[i]);
            acc0 = __builtin_amdgcn_mfma_f32_16x16x32_bf16(__builtin_bit_cast(v8bf, wf0), xa, acc0, 0, 0, 0);
            acc1 = __builtin_amdgcn_mfma_f32_16x16x32_bf16(__builtin_bit_cast(v8bf, wf1), xa, acc1, 0, 0, 0);
        }
    };
    if (foreign) {
        const char* st = myws + WS_WST + (size_t)(call & 1) * WST_BYTES;
        const int t4 = wave >> 2;
        const char* wl = st + (size_t)(t4 ? slot1 : slot0) * TILE_BYTES + lane * 16;
#pragma unroll
        for (int r = 0; r < 4; ++r)
#pragma unroll
            for (int i = 0; i < 4; ++i) {
                const unsigned blk = 16 * r + 4 * (wave & 3) + i;
                glds16(wl + (size_t)blk * 1024, lds0 + LDS_W + t4 * TILE_BYTES + blk * 1024);
            }
        asm volatile("s_waitcnt vmcnt(0)" ::: "memory");
        if (lane == 0) lds_add(sy + SY_DMA, 1u);
        lds_wait_ge(sy + SY_DMA, 8u);
        if (wave == 0) STAMP(13);
#pragma unroll
        for (int jb = 0; jb < 4; ++jb) issue(jb);
        if (wave == 0) STAMP(6);
#pragma unroll
        for (int jb = 0; jb < 4; ++jb) consume_f(jb);
    } else {
#pragma unroll
        for (int jb = 0; jb < WB; ++jb) {
            const int need = 32 * rtl + 8 * jb + 8 - 8 * WB;
            if (need > 0) lds_wait_ge(done, (unsigned)need);
            issue(jb);
        }
        if (wave == 0) STAMP(6);
        lds_wait_ge(sy + SY_WCNT, 32u);
        if (wave == 0) STAMP(7);
#pragma unroll
        for (int jb = 0; jb < 4; ++jb) {
            consume(jb);
            ORDER();
            if (lane == 0) lds_max(done, 32 * rtl + 8 * jb + 8);
            if (jb + WB < 4) issue(jb + WB);
        }
    }
    if (wave == 0) STAMP(8);
    if (wave == 7) STAMP(9);
    {
        char* red = lds + LDS_RED + rtl * (2 * 16 * RED_ROW);
        char* mine = red + kh * (16 * RED_ROW) + (lane & 15) * RED_ROW + (lane >> 4) * 16;
        *(v4f*)mine = acc0;
        *(v4f*)(mine + 64) = acc1;
        if (lane == 0) lds_st(sy + SY_PAIR + wave * 4, 1u);
        lds_wait_ge(sy + SY_PAIR + (wave ^ 1) * 4, 1u);
        const int row = kh * 8 + (lane >> 3), c4 = lane & 7;
        const v4f s0 = *(const v4f*)(red + row * RED_ROW + c4 * 16);
        const v4f s1 = *(const v4f*)(red + 16 * RED_ROW + row * RED_ROW + c4 * 16);
        const v4f y = s0 + s1;
        v2u o;
        if (!bad) { o.x = pack_bf16(silu(y[0]), silu(y[1])); o.y = pack_bf16(silu(y[2]), silu(y[3])); }
        else { o.x = o.y = 0x7FC07FC0u; }
        const int grow = src * MS + q * 64 + rtl * 16 + row;
        const bool t1 = c4 >= 4;
        char* ob = pick(tab, t1 ? own1 : own0);
        char* op = ob + (size_t)grow * (NS * 2) + (t1 ? col1 : col0) * 2 + (c4 & 3) * 8;
        if (!foreign) {
            *(GAS v2u*)op = o;
        } else {
            asm volatile("global_store_dwordx2 %0, %1, off sc0 sc1\n\ts_waitcnt vmcnt(0)" : : "v"(op), "v"(o) : "memory");
            char* base = pick(tab, 8 + (lane ? own1 : own0));
            if (lane < 2 && (lane == 0 || own1 != own0)) {
                __hip_atomic_store((gu32*)(base + WS_CTRL + CTRL_DFLAG + (size_t)(rank * 1024 + ((q * 4 + mem) * 8 + (cp - 8)) * 8 + wave) * 4), call,
                                   __ATOMIC_RELAXED, __HIP_MEMORY_SCOPE_SYSTEM);
            }
        }
    }
    if (wave == 7) STAMP(10);
}

}

extern "C" void kernel_launch(int rank, int nranks, ncclComm_t comm,
                              hipStream_t stream,
                              void* const* d_in, const int* in_sizes, int n_in,
                              void* d_out, int out_size,
                              const int* in_dtypes, int out_dtype,
                              void* const* d_out_all,
                              void* const* d_ws_all, size_t ws_bytes) {
    if (nranks != NR || n_in != 2 || ws_bytes < WS_NEED || in_sizes[0] != MS * KD || in_sizes[1] != KD * NS ||
        out_size != NR * MS * NS || (unsigned)rank >= (unsigned)NR) {
        fprintf(stderr, "kernel_launch: unexpected problem geometry\n");
        return;
    }
    HIP_CHECK(hipFuncSetAttribute((const void*)ag_gemm_silu, hipFuncAttributeMaxDynamicSharedMemorySize, LDS_BYTES));
    Params p;
    p.x = (const char*)d_in[0];
    p.w = (const char*)d_in[1];
    for (int i = 0; i < NR; ++i) { p.out[i] = (char*)d_out_all[i]; p.ws[i] = (char*)d_ws_all[i]; }
    p.out[rank] = (char*)d_out;
    p.rank = rank;
    hipLaunchKernelGGL(ag_gemm_silu, dim3(NWG), dim3(NTHR), LDS_BYTES, stream, p);
    HIP_CHECK(hipGetLastError());
}
```

```cpp
#include <hip/hip_runtime.h>
#include <rccl/rccl.h>
#include <cstddef>
#include <cstdio>
#include <type_traits>

#define HIP_CHECK(expr) do { hipError_t _e = (expr); if (_e != hipSuccess) { \
    fprintf(stderr, "HIP error %s at %s:%d\n", hipGetErrorString(_e), __FILE__, __LINE__); \
    return; } } while (0)

namespace {

constexpr int NR = 8;
constexpr int KD = 2048;
constexpr int MS = 256;
constexpr int NS = 256;
constexpr int NWG = 256;
constexpr int NCW = 8;
constexpr int NTHR = 768;
#define ZNA 7
constexpr int NA = ZNA, NB = (16 - NA) / 3;
static_assert(NA + 3 * NB == 16, "tile split");
constexpr int NSLOT = NA + 6 * NB;
constexpr unsigned SHARD_BYTES = MS * KD * 2;
constexpr unsigned TILE_BYTES = KD * 32;
constexpr unsigned WST_BYTES = 2u << 20;
static_assert(NSLOT * TILE_BYTES <= WST_BYTES && NSLOT * 128 <= 16384, "staging");
constexpr size_t WS_XCOPY = 0;
constexpr size_t WS_WST = 2ull << 20;
constexpr size_t WS_LL = 8ull << 20;
constexpr size_t WS_CTRL = 16ull << 20;
constexpr size_t CTRL_COUNTER = 0;
constexpr size_t CTRL_TMO = 128;
constexpr size_t CTRL_XFLAG = 4096;
constexpr size_t CTRL_WFLAG = 16384;
constexpr size_t CTRL_DFLAG = 32768;
constexpr size_t WS_NEED = WS_CTRL + CTRL_DFLAG + NR * 1024 * 4;
#define STAMP(i) do { } while (0)
constexpr unsigned long long DEADLINE_TICKS = 20000000ull;

constexpr int LDS_W = 0;
constexpr int RED_ROW = 144;
constexpr int LDS_RED = KD * 64;
constexpr int LDS_SYNC = LDS_RED + 8 * 16 * RED_ROW;
constexpr int SY_XGO = 0, SY_WGO = 4, SY_BAD = 8, SY_WCNT = 12, SY_DMA = 16, SY_SENT = 20, SY_DONE = 24, SY_PAIR = 32, SY_XST = 64, SY_INV = 68, SY_PUSH = 72, SY_WR = 80;
constexpr int LDS_BYTES = LDS_SYNC + 128;
#define XGT 500
#define XGATE (-1)
#define WIN_P 2
#define W_ORDER 0
#define SPLIT_XCD 0
#define PROGRESSIVE 0
#define LLN 0
constexpr int LLW0 = 8 - 2 * LLN;
#define TLB_WARM 0
#define WST_T 0
#define WIN_B 2
#define WPOLL0 450
#define WPOLLT 100
#define XPOLL0 350
#define LLST 1300
constexpr unsigned long long LL_START_TICKS = LLST;
constexpr unsigned long long LOCAL_TICKS = 500;
constexpr unsigned long long XGATE_TICKS = XGT;
#define ZWB 4
constexpr int WB = ZWB;

typedef unsigned v4u __attribute__((ext_vector_type(4)));
typedef unsigned v2u __attribute__((ext_vector_type(2)));
typedef short v8s __attribute__((ext_vector_type(8)));
typedef short v4s __attribute__((ext_vector_type(4)));
typedef float v4f __attribute__((ext_vector_type(4)));
typedef __bf16 v8bf __attribute__((ext_vector_type(8)));
typedef __bf16 v2bf __attribute__((ext_vector_type(2)));
typedef float v2f __attribute__((ext_vector_type(2)));
#define GAS __attribute__((address_space(1)))
#define LAS __attribute__((address_space(3)))
typedef GAS unsigned gu32;

struct Params {
    const char* x;
    const char* w;
    char* out[NR];
    char* ws[NR];
    int rank;
};

__device__ __forceinline__ __amdgpu_buffer_rsrc_t make_rsrc(const void* p, unsigned bytes) {
    const unsigned long long a = (unsigned long long)p;
    const unsigned lo = __builtin_amdgcn_readfirstlane((unsigned)a);
    const unsigned hi = __builtin_amdgcn_readfirstlane((unsigned)(a >> 32));
    void* q = (void*)(((unsigned long long)hi << 32) | lo);
    return __builtin_amdgcn_make_buffer_rsrc(q, 0, bytes, 0x00020000);
}
__device__ __forceinline__ float silu(float y) {
    const float c = fminf(fmaxf(y, -60.0f), 60.0f);
    return y * __builtin_amdgcn_rcpf(1.0f + __expf(-c));
}
__device__ __forceinline__ unsigned pack_bf16(float lo, float hi) {
    v2f f = {lo, hi};
    v2bf h = __builtin_convertvector(f, v2bf);
    return __builtin_bit_cast(unsigned, h);
}
__device__ __forceinline__ int group_off(int q, int m) {
    return q == 3 ? ((m << 1) | ((m ^ (m >> 1)) & 1)) : (((m >> q) << (q + 1)) | (m & ((1 << q) - 1)));
}
__device__ __forceinline__ void foreign_tile(int q, int tf, int& doff, int& slot) {
    if (tf < NA) { doff = 7; slot = tf; return; }
    const int u = tf - NA, bb = u / NB, r = u - bb * NB;
    const int s = (q == 3 || q == bb) ? 0 : 1;
    doff = (1 << bb) ^ (7 * s);
    slot = NA + (2 * bb + s) * NB + r;
}
__device__ __forceinline__ char* pick(unsigned long long tab, int i) {
    const unsigned lo = (unsigned)__builtin_amdgcn_ds_bpermute(i * 4, (int)(unsigned)tab);
    const unsigned hi = (unsigned)__builtin_amdgcn_ds_bpermute(i * 4, (int)(unsigned)(tab >> 32));
    return (char*)(((unsigned long long)hi << 32) | lo);
}

#define ORDER() do { asm volatile("" ::: "memory"); __builtin_amdgcn_sched_barrier(0); } while (0)
#define WAIT_VM0() do { __builtin_amdgcn_s_waitcnt(0x0F70); ORDER(); } while (0)
__device__ __forceinline__ void glds16(const void* gsrc, unsigned lds_dst) {
    unsigned keep;
    asm volatile("s_mov_b32 %0, m0\n\ts_mov_b32 m0, %2\n\ts_nop 0\n\tglobal_load_lds_dwordx4 %1, off\n\ts_mov_b32 m0, %0"
                 : "=&s"(keep) : "v"(gsrc), "s"(lds_dst) : "memory");
}
__device__ __forceinline__ void glds16_sys(const void* gsrc, unsigned lds_dst) {
    unsigned keep;
    asm volatile("s_mov_b32 %0, m0\n\ts_mov_b32 m0, %2\n\ts_nop 0\n\tglobal_load_lds_dwordx4 %1, off sc0 sc1\n\ts_mov_b32 m0, %0"
                 : "=&s"(keep) : "v"(gsrc), "s"(lds_dst) : "memory");
}
__device__ __forceinline__ unsigned lds_ld(unsigned addr) {
    unsigned v; asm volatile("ds_read_b32 %0, %1\n\ts_waitcnt lgkmcnt(0)" : "=v"(v) : "v"(addr) : "memory");
    return __builtin_amdgcn_readfirstlane(v);
}
__device__ __forceinline__ void lds_st(unsigned addr, unsigned v) {
    asm volatile("s_waitcnt lgkmcnt(0)\n\tds_write_b32 %0, %1" : : "v"(addr), "v"(v) : "memory");
}
__device__ __forceinline__ void lds_add(unsigned addr, unsigned v) {
    asm volatile("ds_add_u32 %0, %1" : : "v"(addr), "v"(v) : "memory");
}
__device__ __forceinline__ void lds_max(unsigned addr, unsigned v) {
    asm volatile("ds_max_u32 %0, %1" : : "v"(addr), "v"(v) : "memory");
}
__device__ __forceinline__ unsigned lds_add_rtn(unsigned addr, unsigned v) {
    unsigned r; asm volatile("ds_add_rtn_u32 %0, %1, %2\n\ts_waitcnt lgkmcnt(0)" : "=v"(r) : "v"(addr), "v"(v) : "memory");
    return r;
}
__device__ __forceinline__ void lds_wait_ge(unsigned addr, unsigned want) {
    for (unsigned n = 0; lds_ld(addr) < want && n < (1u << 24); ++n) __builtin_amdgcn_s_sleep(1);
}
__device__ __forceinline__ bool wait_flags(gu32* f, bool need, gu32* f2, bool need2, unsigned call, gu32* tmo, int lane) {
    const unsigned long long t0 = __builtin_amdgcn_s_memrealtime();
    while (true) {
        unsigned v = call, v2 = call;
        if (need) v = __hip_atomic_load(f, __ATOMIC_RELAXED, __HIP_MEMORY_SCOPE_SYSTEM);
        if (need2) v2 = __hip_atomic_load(f2, __ATOMIC_RELAXED, __HIP_MEMORY_SCOPE_SYSTEM);
        unsigned t = __hip_atomic_load(tmo, __ATOMIC_RELAXED, __HIP_MEMORY_SCOPE_AGENT);
        asm volatile("" : "+v"(t), "+v"(v), "+v"(v2));
        if (__all(v >= call && v2 >= call)) break;
        if (__builtin_amdgcn_readfirstlane(t) == call || __builtin_amdgcn_s_memrealtime() - t0 > DEADLINE_TICKS) {
            if (lane == 0) __hip_atomic_store(tmo, call, __ATOMIC_RELAXED, __HIP_MEMORY_SCOPE_AGENT);
            break;
        }
        __builtin_amdgcn_s_sleep(1);
    }
    const unsigned t = __hip_atomic_load(tmo, __ATOMIC_RELAXED, __HIP_MEMORY_SCOPE_AGENT);
    return __builtin_amdgcn_readfirstlane(t) == call;
}
__device__ __forceinline__ bool wait_flags(gu32* f, bool need, unsigned call, gu32* tmo, int lane) {
    return wait_flags(f, need, f, false, call, tmo, lane);
}

__global__ void __launch_bounds__(NTHR, 3) ag_gemm_silu(Params p) {
    extern __shared__ __attribute__((aligned(16))) char lds[];
    const int tid = threadIdx.x, lane = tid & 63;
    const int wave = __builtin_amdgcn_readfirstlane(tid >> 6);
    const int b = blockIdx.x;
    const int rank = p.rank;
    const int g8 = b & 7, j = b >> 3;
    const int q = g8 >> 1, mem = (g8 & 1) * 2 + (j >> 4), cp = j & 15;
    const int src = rank ^ group_off(q, mem);
    const bool foreign = cp >= 8;
    const int rot = (mem == 0 && cp < 8) ? 4 * cp : 0;
    const unsigned long long tab =
        *(const GAS unsigned long long*)((const char*)__builtin_amdgcn_kernarg_segment_ptr() + offsetof(Params, out) + (lane & 15) * 8);
    char* const myws = p.ws[rank];
    gu32* const counter = (gu32*)(myws + WS_CTRL + CTRL_COUNTER);
    gu32* const tmo = (gu32*)(myws + WS_CTRL + CTRL_TMO);
    const unsigned lds0 = (unsigned)(size_t)(LAS char*)lds;
    const unsigned sy = lds0 + LDS_SYNC;

    const unsigned ticket = __hip_atomic_load(counter, __ATOMIC_RELAXED, __HIP_MEMORY_SCOPE_AGENT);

    int own0 = rank, own1 = rank, slot0 = 0, slot1 = 0, col0 = cp * 32, col1 = cp * 32 + 16;
    if (foreign) {
        int d0, d1;
        const int tf = (cp - 8) * 2;
        foreign_tile(q, tf, d0, slot0); foreign_tile(q, tf + 1, d1, slot1);
        own0 = rank ^ d0; own1 = rank ^ d1; col0 = tf * 16; col1 = tf * 16 + 16;
    }
    const int wc = (lane & 3) ^ (((lane >> 5) & 1) << 1);

    if (wave >= NCW) {
        const int h = wave - NCW;
        const unsigned long long tstart = __builtin_amdgcn_s_memrealtime();
        if (h == 0) STAMP(0);
        const int fj = (j >> 4) * 8 + (j & 7);
        v4u v[2 * (NA > NB ? NA : NB) > 4 ? 2 * (NA > NB ? NA : NB) : 4];
        const int pr = cp - 8;
        const int pi = pr < 2 ? pr : pr - 2;
        const bool psend = (pr == 2 || pr == 3);
        const bool isx = !(h & 1);
        const int pc = (fj * 8 + g8) * 2 + (h >> 1);
        const int ji = q * 4 + mem;
        const int u = (pi * 16 + ji) * 2 + (h >> 1);
        const int L = psend ? (h == 1 ? 6 : 7) : u % 6;
        const int wi = psend ? (pr - 2) * 16 + ji : u / 6;
        const int dd = (L == 6) ? 7 : ((1 << (L >> 1)) ^ (7 * (L & 1)));
        const int tile0 = (L == 6) ? 0 : NA + (L >> 1) * NB;
        const int slot0s = (L == 6) ? 0 : NA + L * NB;
        const int nt = (L == 6) ? NA : (L == 7 ? 0 : NB);
        constexpr int NTMAX = NA > NB ? NA : NB;
        if (foreign) {
            if (isx) {
                const char* xs = p.x + (size_t)((pc >> 4) * 16 + (lane & 15)) * (KD * 2) + (pc & 15) * 256 + (lane >> 4) * 16;
#pragma unroll
                for (int f = 0; f < 4; ++f) v[f] = *(const GAS v4u*)(xs + f * 64);
            } else {
                const int pl = lane ^ (((lane >> 4) & 1) << 3);
                const char* wsrc = p.w + (size_t)(wi * 32 + (pl >> 1)) * (NS * 2) + tile0 * 32 + (pl & 1) * 16;
#pragma unroll
                for (int t = 0; t < NTMAX; ++t)
                    if (t < nt) {
                        v[2 * t] = *(const GAS v4u*)(wsrc + t * 32);
                        v[2 * t + 1] = *(const GAS v4u*)(wsrc + (size_t)1024 * (NS * 2) + t * 32);
                    }
            }
        }
        if (h == 0) {
            uint4* z = (uint4*)(lds + LDS_SYNC);
            if (lane < 8) z[lane] = make_uint4(0u, 0u, 0u, 0u);
        }
        ORDER();
        asm volatile("s_waitcnt lgkmcnt(0)" ::: "memory");
        __builtin_amdgcn_s_barrier();
        ORDER();
        const unsigned call = (unsigned)__builtin_amdgcn_readfirstlane(ticket) / NWG + 1;
        if (foreign && isx) {
            const __amdgpu_buffer_rsrc_t rc = make_rsrc(myws + WS_XCOPY + (size_t)(call & 1) * SHARD_BYTES, SHARD_BYTES);
#pragma unroll
            for (int f = 0; f < 4; ++f) __builtin_amdgcn_raw_buffer_store_b128(v[f], rc, pc * 4096 + f * 1024 + lane * 16, 0, 17);
            ORDER();
            if (lane == 0) lds_add(sy + SY_XST, 1u);
            WAIT_VM0();
            if (h == 0 || h == 2) STAMP(1);
            char* base = pick(tab, 8 + (rank ^ group_off(pc >> 6, lane & 3)));
            if (lane < 4)
                __hip_atomic_store((gu32*)(base + WS_CTRL + CTRL_XFLAG + (size_t)(rank * NWG + pc) * 4), call, __ATOMIC_RELAXED, __HIP_MEMORY_SCOPE_SYSTEM);
            ORDER();
        } else if (foreign && nt) {
            char* dws = p.ws[rank ^ dd];
            const __amdgpu_buffer_rsrc_t rd = make_rsrc(dws + WS_WST + (size_t)(call & 1) * WST_BYTES + (size_t)slot0s * TILE_BYTES, (unsigned)nt * TILE_BYTES);
            gu32* fl = (gu32*)(dws + WS_CTRL + CTRL_WFLAG + (size_t)(slot0s * 32 + wi) * 4);
            WAIT_VM0();
            if (h == 1) STAMP(11);
            const unsigned vo = wi * 1024 + lane * 16;
            auto push = [&](auto wn_) {
                constexpr int WN = decltype(wn_)::value;
#pragma unroll
                for (int i = 0; i < 2 * NTMAX; ++i)
                    if (i < 2 * nt) {
                        if (i >= WN) {
                            asm volatile("s_waitcnt vmcnt(%0)" : : "n"(WN - 1) : "memory");
                            if (((i - WN) & 1) && lane == 0)
                                __hip_atomic_store(fl + ((i - WN) >> 1) * 32, call, __ATOMIC_RELAXED, __HIP_MEMORY_SCOPE_SYSTEM);
                            ORDER();
                        }
                        __builtin_amdgcn_raw_buffer_store_b128(v[i], rd, vo + (i & 1) * 32768, (i >> 1) * TILE_BYTES, 17);
                        ORDER();
                    }
#pragma unroll
                for (int k = WN - 1; k >= 0; --k) {
                    const int i = 2 * nt - 1 - k;
                    if ((i & 1) && i >= 0) {
                        if (k == 0) { asm volatile("s_waitcnt vmcnt(%0)" : : "n"((WN - 1) / 2) : "memory"); }
                        else if (k == 2) { asm volatile("s_waitcnt vmcnt(2)" ::: "memory"); }
                        else { asm volatile("s_waitcnt vmcnt(4)" ::: "memory"); }
                        if (lane == 0) __hip_atomic_store(fl + (i >> 1) * 32, call, __ATOMIC_RELAXED, __HIP_MEMORY_SCOPE_SYSTEM);
                        ORDER();
                    }
                }
            };
            if (L == 6) push(std::integral_constant<int, WIN_P>()); else push(std::integral_constant<int, WIN_B>());
            if (h == 1) STAMP(2); else STAMP(12);
        }
        if (foreign && lane == 0) lds_add(sy + SY_PUSH, 1u);
        if (h == 0) {
            if (lane == 0) __hip_atomic_fetch_add(counter, 1u, __ATOMIC_RELAXED, __HIP_MEMORY_SCOPE_AGENT);
            if (foreign) {
                const int sl = (lane & 32) ? slot1 : slot0;
                while (__builtin_amdgcn_s_memrealtime() - tstart < WPOLL0 + (slot1 < NA ? slot1 * WPOLLT : 0)) __builtin_amdgcn_s_sleep(8);
                const bool bad = wait_flags((gu32*)(myws + WS_CTRL + CTRL_WFLAG + (size_t)(sl * 32 + (lane & 31)) * 4), true, call, tmo, lane);
                if (lane == 0) { if (bad) lds_st(sy + SY_BAD, 1u); lds_st(sy + SY_WGO, 1u); }
                STAMP(4);
            }
        } else if (h == 2) {
            bool bad = false;
            unsigned xgo = 1;
            if (src != rank) {
                const int d = src ^ rank, ld = (d == 1) ? 0 : (d == 6) ? 1 : (d == 2) ? 2 : (d == 5) ? 3 : (d == 4) ? 4 : 5;
                while (__builtin_amdgcn_s_memrealtime() - tstart < XPOLL0) __builtin_amdgcn_s_sleep(8);
                bad = wait_flags((gu32*)(myws + WS_CTRL + CTRL_XFLAG + (size_t)(src * NWG + q * 64 + lane) * 4), true,
                                 (gu32*)(myws + WS_CTRL + CTRL_WFLAG + (size_t)((NA + ld * NB + (XGATE >= 0 ? XGATE : 0)) * 32 + (lane & 31)) * 4), XGATE >= 0 && lane < 32,
                                 call, tmo, lane);
                while (__builtin_amdgcn_s_memrealtime() - tstart < XGATE_TICKS) __builtin_amdgcn_s_sleep(1);
            } else {
                gu32* f = (gu32*)(myws + WS_CTRL + CTRL_XFLAG + (size_t)(rank * NWG + q * 64 + lane) * 4);
                xgo = 2;
                do {
                    const unsigned v = __hip_atomic_load(f, __ATOMIC_RELAXED, __HIP_MEMORY_SCOPE_SYSTEM);
                    if (__all(v >= call)) { xgo = 1; break; }
                } while (__builtin_amdgcn_s_memrealtime() - tstart < LOCAL_TICKS);
            }
            if (lane == 0) { if (bad) lds_st(sy + SY_BAD, 1u); lds_st(sy + SY_XGO, xgo); }
            STAMP(3);
        }
        if ((b & 7) != rank) {
            const int s = b & 7, i = b >> 3, jq = i >> 3, jm = (i >> 1) & 3;
            bool bad = false;
            if (LLN > 0) {
                const int pr = (i & 1) * 4 + h, c4 = lane & 7, tf = 2 * pr + (c4 >> 2);
                int d, sl;
                foreign_tile(jq, tf, d, sl);
                const bool need = (d == (s ^ rank));
                if (__any(need)) {
                    const char* blk = myws + WS_LL + (size_t)(((s * 16 + (i >> 1)) * 8 + pr) * (2 * LLN)) * 1024 + lane * 16;
                    char* op = p.out[rank] + (size_t)((s ^ group_off(jq, jm)) * MS + jq * 64 + (LLW0 >> 1) * 16 + (lane >> 3)) * (NS * 2) + tf * 32 + (c4 & 3) * 8;
                    while (__builtin_amdgcn_s_memrealtime() - tstart < LL_START_TICKS) __builtin_amdgcn_s_sleep(16);
                    const unsigned long long t0 = __builtin_amdgcn_s_memrealtime();
                    unsigned pend = (1u << (2 * LLN)) - 1;
                    while (pend) {
                        v4u g[2 * LLN > 0 ? 2 * LLN : 1];
#pragma unroll
                        for (int w = 0; w < 2 * LLN; ++w)
                            asm volatile("global_load_dwordx4 %0, %1, off offset:%2 sc0 sc1" : "=v"(g[w]) : "v"(blk), "n"(w * 1024) : "memory");
                        const unsigned t = __hip_atomic_load(tmo, __ATOMIC_RELAXED, __HIP_MEMORY_SCOPE_AGENT);
                        asm volatile("s_waitcnt vmcnt(0)" ::: "memory");
#pragma unroll
                        for (int w = 0; w < 2 * LLN; ++w) {
                            asm volatile("" : "+v"(g[w]));
                            if ((pend >> w & 1) && __all(!need || (g[w].y == call && g[w].w == call))) {
                                v2u o; o.x = g[w].x; o.y = g[w].z;
                                if (need) *(GAS v2u*)(op + (size_t)w * 8 * (NS * 2)) = o;
                                pend &= ~(1u << w);
                            }
                        }
                        if (!pend) break;
                        if (__builtin_amdgcn_readfirstlane(t) == call || __builtin_amdgcn_s_memrealtime() - t0 > DEADLINE_TICKS) {
                            if (lane == 0) __hip_atomic_store(tmo, call, __ATOMIC_RELAXED, __HIP_MEMORY_SCOPE_AGENT);
                            bad = true;
                            break;
                        }
                        __builtin_amdgcn_s_sleep(1);
                    }
                }
            }
            if (h == 0) {
                const int pr = (i & 1) * 4 + ((lane >> 3) & 3);
                int d0, d1, sl;
                foreign_tile(jq, 2 * pr, d0, sl); foreign_tile(jq, 2 * pr + 1, d1, sl);
                const bool need = lane < 32 && (lane & 7) < LLW0 && (d0 == (s ^ rank) || d1 == (s ^ rank));
                while (__builtin_amdgcn_s_memrealtime() - tstart < LL_START_TICKS) __builtin_amdgcn_s_sleep(16);
                bad = wait_flags((gu32*)(myws + WS_CTRL + CTRL_DFLAG + (size_t)(s * 1024 + i * 32 + (lane & 31)) * 4), need, call, tmo, lane) || bad;
                STAMP(5);
            } else if (h == 1) STAMP(15);
            if (bad && lane == 0) *(GAS unsigned short*)(p.out[rank] + (size_t)rank * MS * (NS * 2)) = 0x7FC0;
        }
        return;
    }

    const int rtl = wave >> 1, kh = wave & 1;
    if (!foreign) {
        const char* wl = p.w + (size_t)(lane >> 2) * (NS * 2) + cp * 64 + wc * 16;
#pragma unroll
        for (int i = 0; i < 16; ++i) {
            const int blk = wave * 16 + i;
            glds16(wl + (size_t)((blk + 2 * rot) & 127) * 16 * (NS * 2), lds0 + LDS_W + blk * 1024);
        }
    }
    ORDER();
    __builtin_amdgcn_s_barrier();
    ORDER();
    const unsigned call = (unsigned)__builtin_amdgcn_readfirstlane(ticket) / NWG + 1;
    if (!foreign) {
        asm volatile("s_waitcnt vmcnt(0)" ::: "memory");
        if (lane == 0) lds_add(sy + SY_WCNT, 4u);
    } else {
        lds_wait_ge(sy + SY_PUSH, 4u);
        lds_wait_ge(sy + SY_WGO, 1u);
    }
    lds_wait_ge(sy + SY_XGO, 1u);
    bool bad = lds_ld(sy + SY_BAD) != 0;
    const bool local = lds_ld(sy + SY_XGO) == 2;
    const char* abase = local ? p.x : (p.ws[src] + WS_XCOPY + (size_t)(call & 1) * SHARD_BYTES);
    const __amdgpu_buffer_rsrc_t ra = make_rsrc(abase, bad ? 0u : SHARD_BYTES);
    const unsigned voff = (q * 4 + rtl) * 65536 + (local ? ((lane & 15) * (KD * 2) + (lane >> 4) * 16) : lane * 16);
    const unsigned kstride = local ? 64u : 1024u;
    v4u a[32];
    v4f acc0 = {0.f, 0.f, 0.f, 0.f}, acc1 = {0.f, 0.f, 0.f, 0.f};
    const unsigned done = sy + SY_DONE + kh * 4;
    auto issue = [&](int jb) {
#pragma unroll
        for (int i = 8 * jb; i < 8 * jb + 8; ++i)
            a[i] = __builtin_amdgcn_raw_buffer_load_b128(ra, voff, ((2 * i + kh + rot) & 63) * kstride, 0);
        ORDER();
    };
    const int g = lane >> 4, r4 = (lane >> 2) & 3, pp = lane & 3;
    const unsigned wlo = LDS_W + kh * 2048 + (8 * g + r4) * 64 + pp * 8;
    const LAS char* w0p = (const LAS char*)lds + wlo + ((g & 1) * 32);
    const LAS char* w1p = (const LAS char*)lds + wlo + (((g & 1) ^ 1) * 32);
    auto consume = [&](int jb) {
#pragma unroll
        for (int i = 8 * jb; i < 8 * jb + 8; ++i) {
            const v4s l0 = __builtin_amdgcn_ds_read_tr16_b64_v4i16((LAS v4s*)(w0p + i * 4096));
            const v4s h0 = __builtin_amdgcn_ds_read_tr16_b64_v4i16((LAS v4s*)(w0p + i * 4096 + 256));
            const v4s l1 = __builtin_amdgcn_ds_read_tr16_b64_v4i16((LAS v4s*)(w1p + i * 4096));
            const v4s h1 = __builtin_amdgcn_ds_read_tr16_b64_v4i16((LAS v4s*)(w1p + i * 4096 + 256));
            const v8s wf0 = __builtin_shufflevector(l0, h0, 0, 1, 2, 3, 4, 5, 6, 7);
            const v8s wf1 = __builtin_shufflevector(l1, h1, 0, 1, 2, 3, 4, 5, 6, 7);
            const v8bf xa = __builtin_bit_cast(v8bf, a[i]);
            acc0 = __builtin_amdgcn_mfma_f32_16x16x32_bf16(__builtin_bit_cast(v8bf, wf0), xa, acc0, 0, 0, 0);
            acc1 = __builtin_amdgcn_mfma_f32_16x16x32_bf16(__builtin_bit_cast(v8bf, wf1), xa, acc1, 0, 0, 0);
        }
    };
    const unsigned flo = LDS_W + kh * 1024 + (8 * g + r4) * 32 + pp * 8;
    const LAS char* f0l = (const LAS char*)lds + flo + ((g & 1) * 128);
    const LAS char* f0h = (const LAS char*)lds + flo + (((g & 1) ^ 1) * 128);
    const LAS char* f1l = f0l + TILE_BYTES;
    const LAS char* f1h = f0h + TILE_BYTES;
    auto consume_f = [&](int jb) {
#pragma unroll
        for (int i = 8 * jb; i < 8 * jb + 8; ++i) {
            const v4s l0 = __builtin_amdgcn_ds_read_tr16_b64_v4i16((LAS v4s*)(f0l + i * 2048));
            const v4s h0 = __builtin_amdgcn_ds_read_tr16_b64_v4i16((LAS v4s*)(f0h + i * 2048));
            const v4s l1 = __builtin_amdgcn_ds_read_tr16_b64_v4i16((LAS v4s*)(f1l + i * 2048));
            const v4s h1 = __builtin_amdgcn_ds_read_tr16_b64_v4i16((LAS v4s*)(f1h + i * 2048));
            const v8s wf0 = __builtin_shufflevector(l0, h0, 0, 1, 2, 3, 4, 5, 6, 7);
            const v8s wf1 = __builtin_shufflevector(l1, h1, 0, 1, 2, 3, 4, 5, 6, 7);
            const v8bf xa = __builtin_bit_cast(v8bf, a[i]);
            acc0 = __builtin_amdgcn_mfma_f32_16x16x32_bf16(__builtin_bit_cast(v8bf, wf0), xa, acc0, 0, 0, 0);
            acc1 = __builtin_amdgcn_mfma_f32_16x16x32_bf16(__builtin_bit_cast(v8bf, wf1), xa, acc1, 0, 0, 0);
        }
    };
    if (foreign) {
        const char* st = myws + WS_WST + (size_t)(call & 1) * WST_BYTES;
        const int t4 = wave >> 2;
        const char* wl = st + (size_t)(t4 ? slot1 : slot0) * TILE_BYTES + lane * 16;
#pragma unroll
        for (int r = 0; r < 4; ++r)
#pragma unroll
            for (int i = 0; i < 4; ++i) {
                const unsigned blk = 16 * r + 4 * (wave & 3) + i;
                glds16(wl + (size_t)blk * 1024, lds0 + LDS_W + t4 * TILE_BYTES + blk * 1024);
            }
        asm volatile("s_waitcnt vmcnt(0)" ::: "memory");
        if (lane == 0) lds_add(sy + SY_DMA, 1u);
        lds_wait_ge(sy + SY_DMA, 8u);
        if (wave == 0) STAMP(13);
#pragma unroll
        for (int jb = 0; jb < 4; ++jb) issue(jb);
        if (wave == 0) STAMP(6);
#pragma unroll
        for (int jb = 0; jb < 4; ++jb) consume_f(jb);
    } else {
#pragma unroll
        for (int jb = 0; jb < WB; ++jb) {
            const int need = 32 * rtl + 8 * jb + 8 - 8 * WB;
            if (need > 0) lds_wait_ge(done, (unsigned)need);
            issue(jb);
        }
        if (wave == 0) STAMP(6);
        lds_wait_ge(sy + SY_WCNT, 32u);
        if (wave == 0) STAMP(7);
#pragma unroll
        for (int jb = 0; jb < 4; ++jb) {
            consume(jb);
            ORDER();
            if (lane == 0) lds_max(done, 32 * rtl + 8 * jb + 8);
            if (jb + WB < 4) issue(jb + WB);
        }
    }
    if (wave == 0) STAMP(8);
    if (wave == 7) STAMP(9);
    {
        char* red = lds + LDS_RED + rtl * (2 * 16 * RED_ROW);
        char* mine = red + kh * (16 * RED_ROW) + (lane & 15) * RED_ROW + (lane >> 4) * 16;
        *(v4f*)mine = acc0;
        *(v4f*)(mine + 64) = acc1;
        if (lane == 0) lds_st(sy + SY_PAIR + wave * 4, 1u);
        lds_wait_ge(sy + SY_PAIR + (wave ^ 1) * 4, 1u);
        const int row = kh * 8 + (lane >> 3), c4 = lane & 7;
        const v4f s0 = *(const v4f*)(red + row * RED_ROW + c4 * 16);
        const v4f s1 = *(const v4f*)(red + 16 * RED_ROW + row * RED_ROW + c4 * 16);
        const v4f y = s0 + s1;
        v2u o;
        if (!bad) { o.x = pack_bf16(silu(y[0]), silu(y[1])); o.y = pack_bf16(silu(y[2]), silu(y[3])); }
        else { o.x = o.y = 0x7FC07FC0u; }
        const int grow = src * MS + q * 64 + rtl * 16 + row;
        const bool t1 = c4 >= 4;
        char* ob = pick(tab, t1 ? own1 : own0);
        char* op = ob + (size_t)grow * (NS * 2) + (t1 ? col1 : col0) * 2 + (c4 & 3) * 8;
        if (!foreign) {
            *(GAS v2u*)op = o;
        } else if (wave >= LLW0) {
            char* base = pick(tab, 8 + (t1 ? own1 : own0));
            char* lp = base + WS_LL + (size_t)((((rank * 16 + q * 4 + mem) * 8 + (cp - 8)) * (2 * LLN)) + (wave - LLW0)) * 1024 + lane * 16;
            v4u gr; gr.x = o.x; gr.y = call; gr.z = o.y; gr.w = call;
            asm volatile("global_store_dwordx4 %0, %1, off sc0 sc1\n\ts_nop 1" : : "v"(lp), "v"(gr) : "memory");
        } else {
            asm volatile("global_store_dwordx2 %0, %1, off sc0 sc1\n\ts_waitcnt vmcnt(0)" : : "v"(op), "v"(o) : "memory");
            char* base = pick(tab, 8 + (lane ? own1 : own0));
            if (lane < 2 && (lane == 0 || own1 != own0)) {
                __hip_atomic_store((gu32*)(base + WS_CTRL + CTRL_DFLAG + (size_t)(rank * 1024 + ((q * 4 + mem) * 8 + (cp - 8)) * 8 + wave) * 4), call,
                                   __ATOMIC_RELAXED, __HIP_MEMORY_SCOPE_SYSTEM);
            }
        }
    }
    if (wave == 7) STAMP(10);
}

}

extern "C" void kernel_launch(int rank, int nranks, ncclComm_t comm,
                              hipStream_t stream,
                              void* const* d_in, const int* in_sizes, int n_in,
                              void* d_out, int out_size,
                              const int* in_dtypes, int out_dtype,
                              void* const* d_out_all,
                              void* const* d_ws_all, size_t ws_bytes) {
    if (nranks != NR || n_in != 2 || ws_bytes < WS_NEED || in_sizes[0] != MS * KD || in_sizes[1] != KD * NS ||
        out_size != NR * MS * NS || (unsigned)rank >= (unsigned)NR) {
        fprintf(stderr, "kernel_launch: unexpected problem geometry\n");
        return;
    }
    HIP_CHECK(hipFuncSetAttribute((const void*)ag_gemm_silu, hipFuncAttributeMaxDynamicSharedMemorySize, LDS_BYTES));
    Params p;
    p.x = (const char*)d_in[0];
    p.w = (const char*)d_in[1];
    for (int i = 0; i < NR; ++i) { p.out[i] = (char*)d_out_all[i]; p.ws[i] = (char*)d_ws_all[i]; }
    p.out[rank] = (char*)d_out;
    p.rank = rank;
    hipLaunchKernelGGL(ag_gemm_silu, dim3(NWG), dim3(NTHR), LDS_BYTES, stream, p);
    HIP_CHECK(hipGetLastError());
}
```

```cpp
#include <hip/hip_runtime.h>
#include <rccl/rccl.h>
#include <cstddef>
#include <cstdio>
#include <type_traits>

#define HIP_CHECK(expr) do { hipError_t _e = (expr); if (_e != hipSuccess) { \
    fprintf(stderr, "HIP error %s at %s:%d\n", hipGetErrorString(_e), __FILE__, __LINE__); \
    return; } } while (0)

namespace {

constexpr int NR = 8;
constexpr int KD = 2048;
constexpr int MS = 256;
constexpr int NS = 256;
constexpr int NWG = 256;
constexpr int NCW = 8;
constexpr int NTHR = 768;
#define ZNA 7
constexpr int NA = ZNA, NB = (16 - NA) / 3;
static_assert(NA + 3 * NB == 16 && NA >= 2 && NB >= 2, "tile split");
constexpr int NSLOT = NA + 6 * NB;
constexpr unsigned SHARD_BYTES = MS * KD * 2;
constexpr unsigned TILE_BYTES = KD * 32;
constexpr unsigned WST_BYTES = 2u << 20;
static_assert(NSLOT * TILE_BYTES <= WST_BYTES && NSLOT * 128 <= 16384, "staging");
constexpr size_t WS_XCOPY = 0;
constexpr size_t WS_WST = 2ull << 20;
constexpr size_t WS_CTRL = 16ull << 20;
constexpr size_t CTRL_COUNTER = 0;
constexpr size_t CTRL_TMO = 128;
constexpr size_t CTRL_XFLAG = 4096;
constexpr size_t CTRL_WFLAG = 16384;
constexpr size_t CTRL_DFLAG = 32768;
constexpr size_t WS_NEED = WS_CTRL + CTRL_DFLAG + NR * 1024 * 4;
#define STAMP(i) do { } while (0)

constexpr unsigned long long DEADLINE_TICKS = 20000000ull;
#define XGT 500
constexpr unsigned long long XGATE_TICKS = XGT;
constexpr unsigned long long XPOLL_TICKS = 350;
constexpr unsigned long long WPOLL_TICKS = 450, WPOLL_TILE_TICKS = 100;
constexpr unsigned long long DPOLL_TICKS = 1300;
constexpr unsigned long long LOCAL_TICKS = 500;
#define ZWB 4
constexpr int WB = ZWB;
#define WIN_P 2
#define WIN_B 2

constexpr int LDS_W = 0;
constexpr int RED_ROW = 144;
constexpr int LDS_RED = KD * 64;
constexpr int LDS_SYNC = LDS_RED + 8 * 16 * RED_ROW;
constexpr int SY_XGO = 0, SY_WGO = 4, SY_BAD = 8, SY_WCNT = 12, SY_COMM = 16, SY_DONE = 24, SY_PAIR = 32;
constexpr int LDS_BYTES = LDS_SYNC + 64;

typedef unsigned v4u __attribute__((ext_vector_type(4)));
typedef unsigned v2u __attribute__((ext_vector_type(2)));
typedef short v8s __attribute__((ext_vector_type(8)));
typedef short v4s __attribute__((ext_vector_type(4)));
typedef float v4f __attribute__((ext_vector_type(4)));
typedef __bf16 v8bf __attribute__((ext_vector_type(8)));
typedef __bf16 v2bf __attribute__((ext_vector_type(2)));
typedef float v2f __attribute__((ext_vector_type(2)));
#define GAS __attribute__((address_space(1)))
#define LAS __attribute__((address_space(3)))
typedef GAS unsigned gu32;

struct Params {
    const char* x;
    const char* w;
    char* out[NR];
    char* ws[NR];
    char* myws;
    int rank;
};

__device__ __forceinline__ __amdgpu_buffer_rsrc_t make_rsrc(const void* p, unsigned bytes) {
    const unsigned long long a = (unsigned long long)p;
    const unsigned lo = __builtin_amdgcn_readfirstlane((unsigned)a);
    const unsigned hi = __builtin_amdgcn_readfirstlane((unsigned)(a >> 32));
    void* q = (void*)(((unsigned long long)hi << 32) | lo);
    return __builtin_amdgcn_make_buffer_rsrc(q, 0, bytes, 0x00020000);
}
__device__ __forceinline__ float silu(float y) {
    const float c = fminf(fmaxf(y, -60.0f), 60.0f);
    return y * __builtin_amdgcn_rcpf(1.0f + __expf(-c));
}
__device__ __forceinline__ unsigned pack_bf16(float lo, float hi) {
    v2f f = {lo, hi};
    v2bf h = __builtin_convertvector(f, v2bf);
    return __builtin_bit_cast(unsigned, h);
}
__device__ __forceinline__ int group_off(int q, int m) {
    return q == 3 ? ((m << 1) | ((m ^ (m >> 1)) & 1)) : (((m >> q) << (q + 1)) | (m & ((1 << q) - 1)));
}
__device__ __forceinline__ void foreign_tile(int q, int tf, int& doff, int& slot) {
    if (tf < NA) { doff = 7; slot = tf; return; }
    const int u = tf - NA, bb = u / NB, r = u - bb * NB;
    const int s = (q == 3 || q == bb) ? 0 : 1;
    doff = (1 << bb) ^ (7 * s);
    slot = NA + (2 * bb + s) * NB + r;
}
__device__ __forceinline__ char* pick(unsigned long long tab, int i) {
    const unsigned lo = (unsigned)__builtin_amdgcn_ds_bpermute(i * 4, (int)(unsigned)tab);
    const unsigned hi = (unsigned)__builtin_amdgcn_ds_bpermute(i * 4, (int)(unsigned)(tab >> 32));
    return (char*)(((unsigned long long)hi << 32) | lo);
}

#define ORDER() do { asm volatile("" ::: "memory"); __builtin_amdgcn_sched_barrier(0); } while (0)
#define WAIT_VM0() do { __builtin_amdgcn_s_waitcnt(0x0F70); ORDER(); } while (0)
__device__ __forceinline__ unsigned long long now() { return __builtin_amdgcn_s_memrealtime(); }
__device__ __forceinline__ unsigned ld_sys(gu32* p) { return __hip_atomic_load(p, __ATOMIC_RELAXED, __HIP_MEMORY_SCOPE_SYSTEM); }
__device__ __forceinline__ void st_sys(gu32* p, unsigned v) { __hip_atomic_store(p, v, __ATOMIC_RELAXED, __HIP_MEMORY_SCOPE_SYSTEM); }
__device__ __forceinline__ void glds16(const void* gsrc, unsigned lds_dst) {
    unsigned keep;
    asm volatile("s_mov_b32 %0, m0\n\ts_mov_b32 m0, %2\n\ts_nop 0\n\tglobal_load_lds_dwordx4 %1, off\n\ts_mov_b32 m0, %0"
                 : "=&s"(keep) : "v"(gsrc), "s"(lds_dst) : "memory");
}
__device__ __forceinline__ unsigned lds_ld(unsigned addr) {
    unsigned v; asm volatile("ds_read_b32 %0, %1\n\ts_waitcnt lgkmcnt(0)" : "=v"(v) : "v"(addr) : "memory");
    return __builtin_amdgcn_readfirstlane(v);
}
__device__ __forceinline__ void lds_st(unsigned addr, unsigned v) {
    asm volatile("s_waitcnt lgkmcnt(0)\n\tds_write_b32 %0, %1" : : "v"(addr), "v"(v) : "memory");
}
__device__ __forceinline__ void lds_add(unsigned addr, unsigned v) {
    asm volatile("ds_add_u32 %0, %1" : : "v"(addr), "v"(v) : "memory");
}
__device__ __forceinline__ void lds_max(unsigned addr, unsigned v) {
    asm volatile("ds_max_u32 %0, %1" : : "v"(addr), "v"(v) : "memory");
}
__device__ __forceinline__ void lds_wait_ge(unsigned addr, unsigned want) {
    for (unsigned n = 0; lds_ld(addr) < want && n < (1u << 24); ++n) __builtin_amdgcn_s_sleep(1);
}
__device__ __forceinline__ bool wait_flags(gu32* f, bool need, unsigned call, gu32* tmo, int lane) {
    const unsigned long long t0 = now();
    while (true) {
        unsigned v = call;
        if (need) v = ld_sys(f);
        unsigned t = __hip_atomic_load(tmo, __ATOMIC_RELAXED, __HIP_MEMORY_SCOPE_AGENT);
        asm volatile("" : "+v"(t), "+v"(v));
        if (__all(v >= call)) break;
        if (__builtin_amdgcn_readfirstlane(t) == call || now() - t0 > DEADLINE_TICKS) {
            if (lane == 0) __hip_atomic_store(tmo, call, __ATOMIC_RELAXED, __HIP_MEMORY_SCOPE_AGENT);
            break;
        }
        __builtin_amdgcn_s_sleep(1);
    }
    const unsigned t = __hip_atomic_load(tmo, __ATOMIC_RELAXED, __HIP_MEMORY_SCOPE_AGENT);
    return __builtin_amdgcn_readfirstlane(t) == call;
}

__global__ void __launch_bounds__(NTHR, 3) ag_gemm_silu(Params p) {
    extern __shared__ __attribute__((aligned(16))) char lds[];
    const int tid = threadIdx.x, lane = tid & 63;
    const int wave = __builtin_amdgcn_readfirstlane(tid >> 6);
    const int b = blockIdx.x;
    const int g8 = b & 7, j = b >> 3;
    const int q = g8 >> 1, mem = (g8 & 1) * 2 + (j >> 4), cp = j & 15;
    const bool foreign = cp >= 8;
    const int rank = p.rank;
    char* const myws = p.myws;
    const char* px = p.x;
    const char* pw = p.w;
    asm volatile("" : "+s"(px), "+s"(pw));
    gu32* const counter = (gu32*)(myws + WS_CTRL + CTRL_COUNTER);
    gu32* const tmo = (gu32*)(myws + WS_CTRL + CTRL_TMO);
    const unsigned lds0 = (unsigned)(size_t)(LAS char*)lds;
    const unsigned sy = lds0 + LDS_SYNC;

    const unsigned ticket = __hip_atomic_load(counter, __ATOMIC_RELAXED, __HIP_MEMORY_SCOPE_AGENT);

    if (wave >= NCW) {
        const int h = wave - NCW;
        const unsigned long long tstart = now();
        const int pr = cp - 8, ji = q * 4 + mem;
        const bool isx = !(h & 1);
        const int pc = ((((j >> 4) * 8 + (j & 7)) * 8 + g8) * 2 + (h >> 1)) & 255;
        const int u = (((pr < 2 ? pr : pr - 2) * 16 + ji) * 2 + (h >> 1));
        const bool psend = (pr == 2 || pr == 3);
        const int L = psend ? (h == 1 ? 6 : 7) : u % 6;
        const int wi = psend ? (pr - 2) * 16 + ji : u / 6;
        const int tile0 = (L == 6) ? 0 : NA + (L >> 1) * NB;
        const int nt = (L == 6) ? NA : (L == 7 ? 0 : NB);
        constexpr int NTMAX = NA > NB ? NA : NB;
        v4u v[2 * NTMAX];
        if (foreign) {
            if (isx) {
                const char* xs = px + (size_t)((pc >> 4) * 16 + (lane & 15)) * (KD * 2) + (pc & 15) * 256 + (lane >> 4) * 16;
#pragma unroll
                for (int f = 0; f < 4; ++f) v[f] = *(const GAS v4u*)(xs + f * 64);
            } else {
                const int pl = lane ^ (((lane >> 4) & 1) << 3);
                const char* wsrc = pw + (size_t)(wi * 32 + (pl >> 1)) * (NS * 2) + tile0 * 32 + (pl & 1) * 16;
#pragma unroll
                for (int t = 0; t < NTMAX; ++t)
                    if (t < nt) {
                        v[2 * t] = *(const GAS v4u*)(wsrc + t * 32);
                        v[2 * t + 1] = *(const GAS v4u*)(wsrc + (size_t)1024 * (NS * 2) + t * 32);
                    }
            }
        }
        ORDER();
        const unsigned long long tab =
            *(const GAS unsigned long long*)((const char*)__builtin_amdgcn_kernarg_segment_ptr() + offsetof(Params, out) + (lane & 15) * 8);
        if (h == 0) {
            STAMP(0);
            uint4* z = (uint4*)(lds + LDS_SYNC);
            if (lane < 4) z[lane] = make_uint4(0u, 0u, 0u, 0u);
        }
        ORDER();
        asm volatile("s_waitcnt lgkmcnt(0)" ::: "memory");
        __builtin_amdgcn_s_barrier();
        ORDER();
        const unsigned call = (unsigned)__builtin_amdgcn_readfirstlane(ticket) / NWG + 1;
        if (foreign && isx) {
            const __amdgpu_buffer_rsrc_t rc = make_rsrc(myws + WS_XCOPY + (size_t)(call & 1) * SHARD_BYTES, SHARD_BYTES);
#pragma unroll
            for (int f = 0; f < 4; ++f) __builtin_amdgcn_raw_buffer_store_b128(v[f], rc, pc * 4096 + f * 1024 + lane * 16, 0, 17);
            ORDER();
            char* base = pick(tab, 8 + (rank ^ group_off(pc >> 6, lane & 3)));
            WAIT_VM0();
            if (h == 0) STAMP(1);
            if (lane < 4) st_sys((gu32*)(base + WS_CTRL + CTRL_XFLAG + (size_t)(rank * NWG + pc) * 4), call);
            ORDER();
        } else if (foreign && nt) {
            const int dd = (L == 6) ? 7 : ((1 << (L >> 1)) ^ (7 * (L & 1)));
            const int slot0s = (L == 6) ? 0 : NA + L * NB;
            char* dws = p.ws[rank ^ dd];
            const __amdgpu_buffer_rsrc_t rd = make_rsrc(dws + WS_WST + (size_t)(call & 1) * WST_BYTES + (size_t)slot0s * TILE_BYTES, (unsigned)nt * TILE_BYTES);
            gu32* fl = (gu32*)(dws + WS_CTRL + CTRL_WFLAG + (size_t)(slot0s * 32 + wi) * 4);
            WAIT_VM0();
            if (h == 1) STAMP(11);
            const unsigned vo = wi * 1024 + lane * 16;
            auto push = [&](auto wn_) {
                constexpr int WN = decltype(wn_)::value;
                static_assert(WN >= 1 && WN <= 4, "window");
#pragma unroll
                for (int i = 0; i < 2 * NTMAX; ++i)
                    if (i < 2 * nt) {
                        if (i >= WN) {
                            asm volatile("s_waitcnt vmcnt(%0)" : : "n"(WN - 1) : "memory");
                            if (((i - WN) & 1) && lane == 0) st_sys(fl + ((i - WN) >> 1) * 32, call);
                            ORDER();
                        }
                        __builtin_amdgcn_raw_buffer_store_b128(v[i], rd, vo + (i & 1) * 32768, (i >> 1) * TILE_BYTES, 17);
                        ORDER();
                    }
                if (WN >= 3) {
                    asm volatile("s_waitcnt vmcnt(2)" ::: "memory");
                    if (lane == 0) st_sys(fl + (nt - 2) * 32, call);
                    ORDER();
                    asm volatile("s_waitcnt vmcnt(1)" ::: "memory");
                } else {
                    asm volatile("s_waitcnt vmcnt(0)" ::: "memory");
                }
                if (lane == 0) st_sys(fl + (nt - 1) * 32, call);
            };
            if (L == 6) push(std::integral_constant<int, WIN_P>()); else push(std::integral_constant<int, WIN_B>());
            if (h == 1) STAMP(2); else STAMP(12);
        }
        if (foreign && lane == 0) lds_add(sy + SY_COMM, 1u);
        if (h == 0) {
            if (lane == 0) __hip_atomic_fetch_add(counter, 1u, __ATOMIC_RELAXED, __HIP_MEMORY_SCOPE_AGENT);
            if (foreign) {
                int d0, d1, slot0, slot1;
                foreign_tile(q, 2 * pr, d0, slot0); foreign_tile(q, 2 * pr + 1, d1, slot1);
                gu32* wf = (gu32*)(myws + WS_CTRL + CTRL_WFLAG + (size_t)(((lane & 32) ? slot1 : slot0) * 32 + (lane & 31)) * 4);
                if (!__all(ld_sys(wf) >= call))
                    while (now() - tstart < WPOLL_TICKS + (slot1 < NA ? slot1 * WPOLL_TILE_TICKS : 0)) __builtin_amdgcn_s_sleep(8);
                const bool bad = wait_flags(wf, true, call, tmo, lane);
                if (lane == 0) { if (bad) lds_st(sy + SY_BAD, 1u); lds_st(sy + SY_WGO, 1u); }
                STAMP(4);
            }
            if (g8 != rank) {
                const int pp = (j & 1) * 4 + ((lane >> 3) & 3);
                int d0, d1, sl;
                foreign_tile(j >> 3, 2 * pp, d0, sl); foreign_tile(j >> 3, 2 * pp + 1, d1, sl);
                const bool need = lane < 32 && (d0 == (g8 ^ rank) || d1 == (g8 ^ rank));
                while (now() - tstart < DPOLL_TICKS) __builtin_amdgcn_s_sleep(16);
                const bool bad = wait_flags((gu32*)(myws + WS_CTRL + CTRL_DFLAG + (size_t)(g8 * 1024 + j * 32 + (lane & 31)) * 4), need, call, tmo, lane);
                if (bad && lane == 0) *(GAS unsigned short*)(p.out[rank] + (size_t)rank * MS * (NS * 2)) = 0x7FC0;
                STAMP(5);
            }
        } else if (h == 2) {
            const int d = group_off(q, mem), src = rank ^ d;
            gu32* xf = (gu32*)(myws + WS_CTRL + CTRL_XFLAG + (size_t)(src * NWG + q * 64 + lane) * 4);
            bool bad = false;
            unsigned xgo = 1;
            if (d != 0) {
                if (!__all(ld_sys(xf) >= call))
                    while (now() - tstart < XPOLL_TICKS) __builtin_amdgcn_s_sleep(8);
                bad = wait_flags(xf, true, call, tmo, lane);
                const int ld = (d == 1) ? 0 : (d == 6) ? 1 : (d == 2) ? 2 : (d == 5) ? 3 : (d == 4) ? 4 : 5;
                gu32* wf = (gu32*)(myws + WS_CTRL + CTRL_WFLAG + (size_t)((NA + ld * NB + NB - 1) * 32 + (lane & 31)) * 4);
                while (now() - tstart < XGATE_TICKS) {
                    if (__all(ld_sys(wf) >= call)) break;
                    __builtin_amdgcn_s_sleep(4);
                }
            } else {
                xgo = 2;
                do {
                    if (__all(ld_sys(xf) >= call)) { xgo = 1; break; }
                } while (now() - tstart < LOCAL_TICKS);
            }
            if (lane == 0) { if (bad) lds_st(sy + SY_BAD, 1u); lds_st(sy + SY_XGO, xgo); }
            STAMP(3);
        }
        return;
    }

    const int rtl = wave >> 1, kh = wave & 1;
    const int src = rank ^ group_off(q, mem);
    const int rot = (mem == 0 && !foreign) ? 4 * cp : 0;
    if (!foreign) {
        const int wc = (lane & 3) ^ (((lane >> 5) & 1) << 1);
        const char* wl = pw + (size_t)(lane >> 2) * (NS * 2) + cp * 64 + wc * 16;
#pragma unroll
        for (int i = 0; i < 16; ++i) {
            const int blk = wave * 16 + i;
            glds16(wl + (size_t)((blk + 2 * rot) & 127) * 16 * (NS * 2), lds0 + LDS_W + blk * 1024);
        }
    }
    ORDER();
    const unsigned long long tab =
        *(const GAS unsigned long long*)((const char*)__builtin_amdgcn_kernarg_segment_ptr() + offsetof(Params, out) + (lane & 15) * 8);
    ORDER();
    __builtin_amdgcn_s_barrier();
    ORDER();
    const unsigned call = (unsigned)__builtin_amdgcn_readfirstlane(ticket) / NWG + 1;
    int own0 = rank, own1 = rank, slot0 = 0, slot1 = 0, col0 = cp * 32, col1 = cp * 32 + 16;
    if (!foreign) {
        asm volatile("s_waitcnt vmcnt(0)" ::: "memory");
        if (lane == 0) lds_add(sy + SY_WCNT, 1u);
    } else {
        int d0, d1;
        const int tf = (cp - 8) * 2;
        foreign_tile(q, tf, d0, slot0); foreign_tile(q, tf + 1, d1, slot1);
        own0 = rank ^ d0; own1 = rank ^ d1; col0 = tf * 16; col1 = tf * 16 + 16;
        lds_wait_ge(sy + SY_COMM, 4u);
        lds_wait_ge(sy + SY_WGO, 1u);
        const int t4 = wave >> 2;
        const char* wl = myws + WS_WST + (size_t)(call & 1) * WST_BYTES + (size_t)(t4 ? slot1 : slot0) * TILE_BYTES + lane * 16;
#pragma unroll
        for (int i = 0; i < 16; ++i) {
            const unsigned blk = 16 * (wave & 3) + i;
            glds16(wl + (size_t)blk * 1024, lds0 + LDS_W + t4 * TILE_BYTES + blk * 1024);
        }
        asm volatile("s_waitcnt vmcnt(0)" ::: "memory");
        if (lane == 0) lds_add(sy + SY_WCNT, 1u);
    }
    const bool t1 = (lane & 7) >= 4;
    char* op = pick(tab, t1 ? own1 : own0) + (size_t)(src * MS + q * 64 + rtl * 16 + kh * 8 + (lane >> 3)) * (NS * 2) + (t1 ? col1 : col0) * 2 + (lane & 3) * 8;
    char* fp = pick(tab, 8 + ((lane & 1) ? own1 : own0)) + WS_CTRL + CTRL_DFLAG + (size_t)(rank * 1024 + ((q * 4 + mem) * 8 + (cp & 7)) * 8 + wave) * 4;
    lds_wait_ge(sy + SY_XGO, 1u);
    const bool local = lds_ld(sy + SY_XGO) == 2;
    const bool bad = lds_ld(sy + SY_BAD) != 0;
    const char* abase = local ? px : (p.ws[src] + WS_XCOPY + (size_t)(call & 1) * SHARD_BYTES);
    const __amdgpu_buffer_rsrc_t ra = make_rsrc(abase, bad ? 0u : SHARD_BYTES);
    const unsigned voff = (q * 4 + rtl) * 65536 + (local ? ((lane & 15) * (KD * 2) + (lane >> 4) * 16) : lane * 16);
    const unsigned kstride = local ? 64u : 1024u;
    v4u a[32];
    v4f acc0 = {0.f, 0.f, 0.f, 0.f}, acc1 = {0.f, 0.f, 0.f, 0.f};
    auto issue = [&](int jb) {
#pragma unroll
        for (int i = 8 * jb; i < 8 * jb + 8; ++i)
            a[i] = __builtin_amdgcn_raw_buffer_load_b128(ra, voff, ((2 * i + kh + rot) & 63) * kstride, 0);
        ORDER();
    };
    const int g = lane >> 4, r4 = (lane >> 2) & 3, c4 = lane & 3;
    auto mma = [&](int i, const LAS char* p0l, const LAS char* p0h, const LAS char* p1l, const LAS char* p1h) {
        const v4s l0 = __builtin_amdgcn_ds_read_tr16_b64_v4i16((LAS v4s*)p0l);
        const v4s h0 = __builtin_amdgcn_ds_read_tr16_b64_v4i16((LAS v4s*)p0h);
        const v4s l1 = __builtin_amdgcn_ds_read_tr16_b64_v4i16((LAS v4s*)p1l);
        const v4s h1 = __builtin_amdgcn_ds_read_tr16_b64_v4i16((LAS v4s*)p1h);
        const v8s wf0 = __builtin_shufflevector(l0, h0, 0, 1, 2, 3, 4, 5, 6, 7);
        const v8s wf1 = __builtin_shufflevector(l1, h1, 0, 1, 2, 3, 4, 5, 6, 7);
        const v8bf xa = __builtin_bit_cast(v8bf, a[i]);
        acc0 = __builtin_amdgcn_mfma_f32_16x16x32_bf16(__builtin_bit_cast(v8bf, wf0), xa, acc0, 0, 0, 0);
        acc1 = __builtin_amdgcn_mfma_f32_16x16x32_bf16(__builtin_bit_cast(v8bf, wf1), xa, acc1, 0, 0, 0);
    };
    if (foreign) {
        lds_wait_ge(sy + SY_WCNT, 8u);
        if (wave == 0) STAMP(13);
#pragma unroll
        for (int jb = 0; jb < 4; ++jb) issue(jb);
        if (wave == 0) STAMP(6);
        const unsigned flo = LDS_W + kh * 1024 + (8 * g + r4) * 32 + c4 * 8;
        const LAS char* f0l = (const LAS char*)lds + flo + ((g & 1) * 128);
        const LAS char* f0h = (const LAS char*)lds + flo + (((g & 1) ^ 1) * 128);
        const LAS char* f1l = f0l + TILE_BYTES;
        const LAS char* f1h = f0h + TILE_BYTES;
#pragma unroll
        for (int i = 0; i < 32; ++i) mma(i, f0l + i * 2048, f0h + i * 2048, f1l + i * 2048, f1h + i * 2048);
    } else {
        const unsigned done = sy + SY_DONE + kh * 4;
#pragma unroll
        for (int jb = 0; jb < WB; ++jb) {
            const int need = 32 * rtl + 8 * jb + 8 - 8 * WB;
            if (need > 0) lds_wait_ge(done, (unsigned)need);
            issue(jb);
        }
        if (wave == 0) STAMP(6);
        lds_wait_ge(sy + SY_WCNT, 8u);
        const unsigned wlo = LDS_W + kh * 2048 + (8 * g + r4) * 64 + c4 * 8;
        const LAS char* w0p = (const LAS char*)lds + wlo + ((g & 1) * 32);
        const LAS char* w1p = (const LAS char*)lds + wlo + (((g & 1) ^ 1) * 32);
#pragma unroll
        for (int jb = 0; jb < 4; ++jb) {
#pragma unroll
            for (int i = 8 * jb; i < 8 * jb + 8; ++i) mma(i, w0p + i * 4096, w0p + i * 4096 + 256, w1p + i * 4096, w1p + i * 4096 + 256);
            ORDER();
            if (lane == 0) lds_max(done, 32 * rtl + 8 * jb + 8);
            if (jb + WB < 4) issue(jb + WB);
        }
    }
    if (wave == 0) STAMP(8);
    if (wave == 7) STAMP(9);
    {
        char* red = lds + LDS_RED + rtl * (2 * 16 * RED_ROW);
        char* mine = red + kh * (16 * RED_ROW) + (lane & 15) * RED_ROW + (lane >> 4) * 16;
        *(v4f*)mine = acc0;
        *(v4f*)(mine + 64) = acc1;
        if (lane == 0) lds_st(sy + SY_PAIR + wave * 4, 1u);
        lds_wait_ge(sy + SY_PAIR + (wave ^ 1) * 4, 1u);
        const int row = kh * 8 + (lane >> 3);
        const v4f s0 = *(const v4f*)(red + row * RED_ROW + (lane & 7) * 16);
        const v4f s1 = *(const v4f*)(red + 16 * RED_ROW + row * RED_ROW + (lane & 7) * 16);
        const v4f y = s0 + s1;
        v2u o;
        if (!bad) { o.x = pack_bf16(silu(y[0]), silu(y[1])); o.y = pack_bf16(silu(y[2]), silu(y[3])); }
        else { o.x = o.y = 0x7FC07FC0u; }
        if (!foreign) {
            *(GAS v2u*)op = o;
        } else {
            asm volatile("global_store_dwordx2 %0, %1, off sc0 sc1\n\ts_waitcnt vmcnt(0)" : : "v"(op), "v"(o) : "memory");
            if (lane < 2 && (lane == 0 || own1 != own0)) st_sys((gu32*)fp, call);
        }
    }
    if (wave == 7) STAMP(10);
}

}

extern "C" void kernel_launch(int rank, int nranks, ncclComm_t comm,
                              hipStream_t stream,
                              void* const* d_in, const int* in_sizes, int n_in,
                              void* d_out, int out_size,
                              const int* in_dtypes, int out_dtype,
                              void* const* d_out_all,
                              void* const* d_ws_all, size_t ws_bytes) {
    if (nranks != NR || n_in != 2 || ws_bytes < WS_NEED || in_sizes[0] != MS * KD || in_sizes[1] != KD * NS ||
        out_size != NR * MS * NS || (unsigned)rank >= (unsigned)NR) {
        fprintf(stderr, "kernel_launch: unexpected problem geometry\n");
        return;
    }
    HIP_CHECK(hipFuncSetAttribute((const void*)ag_gemm_silu, hipFuncAttributeMaxDynamicSharedMemorySize, LDS_BYTES));
    Params p;
    p.x = (const char*)d_in[0];
    p.w = (const char*)d_in[1];
    for (int i = 0; i < NR; ++i) { p.out[i] = (char*)d_out_all[i]; p.ws[i] = (char*)d_ws_all[i]; }
    p.out[rank] = (char*)d_out;
    p.myws = (char*)d_ws_all[rank];
    p.rank = rank;
    hipLaunchKernelGGL(ag_gemm_silu, dim3(NWG), dim3(NTHR), LDS_BYTES, stream, p);
    HIP_CHECK(hipGetLastError());
}
```

```cpp
#include <hip/hip_runtime.h>
#include <rccl/rccl.h>
#include <cstddef>
#include <cstdio>
#include <type_traits>

#define HIP_CHECK(expr) do { hipError_t _e = (expr); if (_e != hipSuccess) { \
    fprintf(stderr, "HIP error %s at %s:%d\n", hipGetErrorString(_e), __FILE__, __LINE__); \
    return; } } while (0)

namespace {

constexpr int NR = 8;
constexpr int KD = 2048;
constexpr int MS = 256;
constexpr int NS = 256;
constexpr int NWG = 256;
constexpr int NCW = 8;
constexpr int NTHR = 768;
#define ZNA 7
constexpr int NA = ZNA, NB = (16 - NA) / 3;
static_assert(NA + 3 * NB == 16 && NA >= 2 && NB >= 2, "tile split");
constexpr int NSLOT = NA + 6 * NB;
constexpr unsigned SHARD_BYTES = MS * KD * 2;
constexpr unsigned TILE_BYTES = KD * 32;
constexpr unsigned WST_BYTES = 2u << 20;
static_assert(NSLOT * TILE_BYTES <= WST_BYTES && NSLOT * 128 <= 16384, "staging");
constexpr size_t WS_XCOPY = 0;
constexpr size_t WS_WST = 2ull << 20;
constexpr size_t WS_CTRL = 6ull << 20;
constexpr size_t CTRL_COUNTER = 0;
constexpr size_t CTRL_TMO = 128;
constexpr size_t CTRL_XFLAG = 4096;
constexpr size_t CTRL_WFLAG = 16384;
constexpr size_t CTRL_DFLAG = 32768;
constexpr size_t WS_NEED = WS_CTRL + CTRL_DFLAG + NR * 1024 * 4;
#define STAMP(i) do { } while (0)

constexpr unsigned long long DEADLINE_TICKS = 20000000ull;
#define XGT 500
constexpr unsigned long long XGATE_TICKS = XGT;
constexpr unsigned long long XPOLL_TICKS = 350;
constexpr unsigned long long WPOLL_TICKS = 450, WPOLL_TILE_TICKS = 100;
constexpr unsigned long long DPOLL_TICKS = 1300;
constexpr unsigned long long LOCAL_TICKS = 600, LOCAL_F_TICKS = 1500;
#define ZWB 12
constexpr int WB = ZWB;
#define WIN_P 2
#define WIN_B 2

constexpr int LDS_W = 0;
constexpr int RED_ROW = 144;
constexpr int LDS_RED = KD * 64;
constexpr int LDS_SYNC = LDS_RED + 8 * 16 * RED_ROW;
constexpr int SY_XGO = 0, SY_WGO = 4, SY_BAD = 8, SY_WCNT = 12, SY_COMM = 16, SY_DONE = 24, SY_PAIR = 32;
constexpr int LDS_BYTES = LDS_SYNC + 64;

typedef unsigned v4u __attribute__((ext_vector_type(4)));
typedef unsigned v2u __attribute__((ext_vector_type(2)));
typedef short v8s __attribute__((ext_vector_type(8)));
typedef short v4s __attribute__((ext_vector_type(4)));
typedef float v4f __attribute__((ext_vector_type(4)));
typedef __bf16 v8bf __attribute__((ext_vector_type(8)));
typedef __bf16 v2bf __attribute__((ext_vector_type(2)));
typedef float v2f __attribute__((ext_vector_type(2)));
#define GAS __attribute__((address_space(1)))
#define LAS __attribute__((address_space(3)))
typedef GAS unsigned gu32;

struct Params {
    const char* x;
    const char* w;
    char* out[NR];
    char* ws[NR];
    char* myws;
    int rank;
};

__device__ __forceinline__ __amdgpu_buffer_rsrc_t make_rsrc(const void* p, unsigned bytes) {
    const unsigned long long a = (unsigned long long)p;
    const unsigned lo = __builtin_amdgcn_readfirstlane((unsigned)a);
    const unsigned hi = __builtin_amdgcn_readfirstlane((unsigned)(a >> 32));
    void* q = (void*)(((unsigned long long)hi << 32) | lo);
    return __builtin_amdgcn_make_buffer_rsrc(q, 0, bytes, 0x00020000);
}
__device__ __forceinline__ float silu(float y) {
    const float c = fminf(fmaxf(y, -60.0f), 60.0f);
    return y * __builtin_amdgcn_rcpf(1.0f + __expf(-c));
}
__device__ __forceinline__ unsigned pack_bf16(float lo, float hi) {
    v2f f = {lo, hi};
    v2bf h = __builtin_convertvector(f, v2bf);
    return __builtin_bit_cast(unsigned, h);
}
__device__ __forceinline__ int group_off(int q, int m) {
    return q == 3 ? ((m << 1) | ((m ^ (m >> 1)) & 1)) : (((m >> q) << (q + 1)) | (m & ((1 << q) - 1)));
}
__device__ __forceinline__ void foreign_tile(int q, int tf, int& doff, int& slot) {
    if (tf < NA) { doff = 7; slot = tf; return; }
    const int u = tf - NA, bb = u / NB, r = u - bb * NB;
    const int s = (q == 3 || q == bb) ? 0 : 1;
    doff = (1 << bb) ^ (7 * s);
    slot = NA + (2 * bb + s) * NB + r;
}
__device__ __forceinline__ char* pick(unsigned long long tab, int i) {
    const unsigned lo = (unsigned)__builtin_amdgcn_ds_bpermute(i * 4, (int)(unsigned)tab);
    const unsigned hi = (unsigned)__builtin_amdgcn_ds_bpermute(i * 4, (int)(unsigned)(tab >> 32));
    return (char*)(((unsigned long long)hi << 32) | lo);
}

#define ORDER() do { asm volatile("" ::: "memory"); __builtin_amdgcn_sched_barrier(0); } while (0)
#define WAIT_VM0() do { __builtin_amdgcn_s_waitcnt(0x0F70); ORDER(); } while (0)
__device__ __forceinline__ unsigned long long now() { return __builtin_amdgcn_s_memrealtime(); }
__device__ __forceinline__ unsigned ld_sys(gu32* p) { return __hip_atomic_load(p, __ATOMIC_RELAXED, __HIP_MEMORY_SCOPE_SYSTEM); }
__device__ __forceinline__ void st_sys(gu32* p, unsigned v) { __hip_atomic_store(p, v, __ATOMIC_RELAXED, __HIP_MEMORY_SCOPE_SYSTEM); }
__device__ __forceinline__ void glds16(const void* gsrc, unsigned lds_dst) {
    unsigned keep;
    asm volatile("s_mov_b32 %0, m0\n\ts_mov_b32 m0, %2\n\ts_nop 0\n\tglobal_load_lds_dwordx4 %1, off\n\ts_mov_b32 m0, %0"
                 : "=&s"(keep) : "v"(gsrc), "s"(lds_dst) : "memory");
}
__device__ __forceinline__ unsigned lds_ld(unsigned addr) {
    unsigned v; asm volatile("ds_read_b32 %0, %1\n\ts_waitcnt lgkmcnt(0)" : "=v"(v) : "v"(addr) : "memory");
    return __builtin_amdgcn_readfirstlane(v);
}
__device__ __forceinline__ void lds_st(unsigned addr, unsigned v) {
    asm volatile("s_waitcnt lgkmcnt(0)\n\tds_write_b32 %0, %1" : : "v"(addr), "v"(v) : "memory");
}
__device__ __forceinline__ void lds_add(unsigned addr, unsigned v) {
    asm volatile("ds_add_u32 %0, %1" : : "v"(addr), "v"(v) : "memory");
}
__device__ __forceinline__ void lds_max(unsigned addr, unsigned v) {
    asm volatile("ds_max_u32 %0, %1" : : "v"(addr), "v"(v) : "memory");
}
__device__ __forceinline__ void lds_wait_ge(unsigned addr, unsigned want) {
    for (unsigned n = 0; lds_ld(addr) < want && n < (1u << 24); ++n) __builtin_amdgcn_s_sleep(1);
}
__device__ __forceinline__ bool wait_flags(gu32* f, bool need, unsigned call, gu32* tmo, int lane) {
    const unsigned long long t0 = now();
    while (true) {
        unsigned v = call;
        if (need) v = ld_sys(f);
        unsigned t = __hip_atomic_load(tmo, __ATOMIC_RELAXED, __HIP_MEMORY_SCOPE_AGENT);
        asm volatile("" : "+v"(t), "+v"(v));
        if (__all(v >= call)) break;
        if (__builtin_amdgcn_readfirstlane(t) == call || now() - t0 > DEADLINE_TICKS) {
            if (lane == 0) __hip_atomic_store(tmo, call, __ATOMIC_RELAXED, __HIP_MEMORY_SCOPE_AGENT);
            break;
        }
        __builtin_amdgcn_s_sleep(1);
    }
    const unsigned t = __hip_atomic_load(tmo, __ATOMIC_RELAXED, __HIP_MEMORY_SCOPE_AGENT);
    return __builtin_amdgcn_readfirstlane(t) == call;
}

__global__ void __launch_bounds__(NTHR, 3) ag_gemm_silu(Params p) {
    extern __shared__ __attribute__((aligned(16))) char lds[];
    const int tid = threadIdx.x, lane = tid & 63;
    const int wave = __builtin_amdgcn_readfirstlane(tid >> 6);
    const int b = blockIdx.x;
    const int g8 = b & 7, j = b >> 3;
    const int q = g8 >> 1, mem = (g8 & 1) * 2 + (j >> 4), cp = j & 15;
    const bool foreign = cp >= 8;
    const int rank = p.rank;
    char* const myws = p.myws;
    const char* px = p.x;
    const char* pw = p.w;
    asm volatile("" : "+s"(px), "+s"(pw));
    gu32* const counter = (gu32*)(myws + WS_CTRL + CTRL_COUNTER);
    gu32* const tmo = (gu32*)(myws + WS_CTRL + CTRL_TMO);
    const unsigned lds0 = (unsigned)(size_t)(LAS char*)lds;
    const unsigned sy = lds0 + LDS_SYNC;

    const unsigned ticket = __hip_atomic_load(counter, __ATOMIC_RELAXED, __HIP_MEMORY_SCOPE_AGENT);

    if (wave >= NCW) {
        const int h = wave - NCW;
        const unsigned long long tstart = now();
        const int pr = cp - 8, ji = q * 4 + mem;
        const bool isx = !(h & 1);
        const int pc = ((((j >> 4) * 8 + (j & 7)) * 8 + g8) * 2 + (h >> 1)) & 255;
        const int u = (((pr < 2 ? pr : pr - 2) * 16 + ji) * 2 + (h >> 1));
        const bool psend = (pr == 2 || pr == 3);
        const int L = psend ? (h == 1 ? 6 : 7) : u % 6;
        const int wi = psend ? (pr - 2) * 16 + ji : u / 6;
        const int tile0 = (L == 6) ? 0 : NA + (L >> 1) * NB;
        const int nt = (L == 6) ? NA : (L == 7 ? 0 : NB);
        constexpr int NTMAX = NA > NB ? NA : NB;
        v4u v[2 * NTMAX];
        if (foreign) {
            if (isx) {
                const char* xs = px + (size_t)((pc >> 4) * 16 + (lane & 15)) * (KD * 2) + (pc & 15) * 256 + (lane >> 4) * 16;
#pragma unroll
                for (int f = 0; f < 4; ++f) v[f] = *(const GAS v4u*)(xs + f * 64);
            } else {
                const int pl = lane ^ (((lane >> 4) & 1) << 3);
                const char* wsrc = pw + (size_t)(wi * 32 + (pl >> 1)) * (NS * 2) + tile0 * 32 + (pl & 1) * 16;
#pragma unroll
                for (int t = 0; t < NTMAX; ++t)
                    if (t < nt) {
                        v[2 * t] = *(const GAS v4u*)(wsrc + t * 32);
                        v[2 * t + 1] = *(const GAS v4u*)(wsrc + (size_t)1024 * (NS * 2) + t * 32);
                    }
            }
        }
        ORDER();
        const unsigned long long tab =
            *(const GAS unsigned long long*)((const char*)__builtin_amdgcn_kernarg_segment_ptr() + offsetof(Params, out) + (lane & 15) * 8);
        if (h == 0) {
            STAMP(0);
            uint4* z = (uint4*)(lds + LDS_SYNC);
            if (lane < 4) z[lane] = make_uint4(0u, 0u, 0u, 0u);
        }
        ORDER();
        asm volatile("s_waitcnt lgkmcnt(0)" ::: "memory");
        __builtin_amdgcn_s_barrier();
        ORDER();
        const unsigned call = (unsigned)__builtin_amdgcn_readfirstlane(ticket) / NWG + 1;
        if (foreign && isx) {
            const __amdgpu_buffer_rsrc_t rc = make_rsrc(myws + WS_XCOPY + (size_t)(call & 1) * SHARD_BYTES, SHARD_BYTES);
#pragma unroll
            for (int f = 0; f < 4; ++f) __builtin_amdgcn_raw_buffer_store_b128(v[f], rc, pc * 4096 + f * 1024 + lane * 16, 0, 17);
            ORDER();
            char* base = pick(tab, 8 + (rank ^ group_off(pc >> 6, lane & 3)));
            WAIT_VM0();
            if (h == 0) STAMP(1);
            if (lane < 4) st_sys((gu32*)(base + WS_CTRL + CTRL_XFLAG + (size_t)(rank * NWG + pc) * 4), call);
            ORDER();
        } else if (foreign && nt) {
            const int dd = (L == 6) ? 7 : ((1 << (L >> 1)) ^ (7 * (L & 1)));
            const int slot0s = (L == 6) ? 0 : NA + L * NB;
            char* dws = p.ws[rank ^ dd];
            const __amdgpu_buffer_rsrc_t rd = make_rsrc(dws + WS_WST + (size_t)(call & 1) * WST_BYTES + (size_t)slot0s * TILE_BYTES, (unsigned)nt * TILE_BYTES);
            gu32* fl = (gu32*)(dws + WS_CTRL + CTRL_WFLAG + (size_t)(slot0s * 32 + wi) * 4);
            WAIT_VM0();
            if (h == 1) STAMP(11);
            const unsigned vo = wi * 1024 + lane * 16;
            auto push = [&](auto wn_) {
                constexpr int WN = decltype(wn_)::value;
                static_assert(WN >= 1 && WN <= 4, "window");
#pragma unroll
                for (int i = 0; i < 2 * NTMAX; ++i)
                    if (i < 2 * nt) {
                        if (i >= WN) {
                            asm volatile("s_waitcnt vmcnt(%0)" : : "n"(WN - 1) : "memory");
                            if (((i - WN) & 1) && lane == 0) st_sys(fl + ((i - WN) >> 1) * 32, call);
                            ORDER();
                        }
                        __builtin_amdgcn_raw_buffer_store_b128(v[i], rd, vo + (i & 1) * 32768, (i >> 1) * TILE_BYTES, 17);
                        ORDER();
                    }
                if (WN >= 3) {
                    asm volatile("s_waitcnt vmcnt(2)" ::: "memory");
                    if (lane == 0) st_sys(fl + (nt - 2) * 32, call);
                    ORDER();
                    asm volatile("s_waitcnt vmcnt(1)" ::: "memory");
                } else {
                    asm volatile("s_waitcnt vmcnt(0)" ::: "memory");
                }
                if (lane == 0) st_sys(fl + (nt - 1) * 32, call);
            };
            if (L == 6) push(std::integral_constant<int, WIN_P>()); else push(std::integral_constant<int, WIN_B>());
            if (h == 1) STAMP(2); else STAMP(12);
        }
        if (foreign && lane == 0) lds_add(sy + SY_COMM, 1u);
        if (h == 0) {
            if (lane == 0) __hip_atomic_fetch_add(counter, 1u, __ATOMIC_RELAXED, __HIP_MEMORY_SCOPE_AGENT);
            if (foreign) {
                int d0, d1, slot0, slot1;
                foreign_tile(q, 2 * pr, d0, slot0); foreign_tile(q, 2 * pr + 1, d1, slot1);
                gu32* wf = (gu32*)(myws + WS_CTRL + CTRL_WFLAG + (size_t)(((lane & 32) ? slot1 : slot0) * 32 + (lane & 31)) * 4);
                if (!__all(ld_sys(wf) >= call))
                    while (now() - tstart < WPOLL_TICKS + (slot1 < NA ? slot1 * WPOLL_TILE_TICKS : 0)) __builtin_amdgcn_s_sleep(8);
                const bool bad = wait_flags(wf, true, call, tmo, lane);
                if (lane == 0) { if (bad) lds_st(sy + SY_BAD, 1u); lds_st(sy + SY_WGO, 1u); }
                STAMP(4);
            }
            if (g8 != rank) {
                const int pp = (j & 1) * 4 + ((lane >> 3) & 3);
                int d0, d1, sl;
                foreign_tile(j >> 3, 2 * pp, d0, sl); foreign_tile(j >> 3, 2 * pp + 1, d1, sl);
                const bool need = lane < 32 && (d0 == (g8 ^ rank) || d1 == (g8 ^ rank));
                while (now() - tstart < DPOLL_TICKS) __builtin_amdgcn_s_sleep(16);
                const bool bad = wait_flags((gu32*)(myws + WS_CTRL + CTRL_DFLAG + (size_t)(g8 * 1024 + j * 32 + (lane & 31)) * 4), need, call, tmo, lane);
                if (bad && lane == 0) *(GAS unsigned short*)(p.out[rank] + (size_t)rank * MS * (NS * 2)) = 0x7FC0;
                STAMP(5);
            }
        } else if (h == 2) {
            const int d = group_off(q, mem), src = rank ^ d;
            gu32* xf = (gu32*)(myws + WS_CTRL + CTRL_XFLAG + (size_t)(src * NWG + q * 64 + lane) * 4);
            bool bad = false;
            unsigned xgo = 1;
            if (d != 0) {
                if (!__all(ld_sys(xf) >= call))
                    while (now() - tstart < XPOLL_TICKS) __builtin_amdgcn_s_sleep(8);
                bad = wait_flags(xf, true, call, tmo, lane);
                const int ld = (d == 1) ? 0 : (d == 6) ? 1 : (d == 2) ? 2 : (d == 5) ? 3 : (d == 4) ? 4 : 5;
                gu32* wf = (gu32*)(myws + WS_CTRL + CTRL_WFLAG + (size_t)((NA + ld * NB + NB - 1) * 32 + (lane & 31)) * 4);
                while (now() - tstart < XGATE_TICKS) {
                    if (__all(ld_sys(wf) >= call)) break;
                    __builtin_amdgcn_s_sleep(4);
                }
            } else {
                xgo = 2;
                do {
                    if (__all(ld_sys(xf) >= call)) { xgo = 1; break; }
                    __builtin_amdgcn_s_sleep(2);
                } while (now() - tstart < (foreign ? LOCAL_F_TICKS : LOCAL_TICKS));
            }
            if (lane == 0) { if (bad) lds_st(sy + SY_BAD, 1u); lds_st(sy + SY_XGO, xgo); }
            STAMP(3);
        }
        return;
    }

    const int rtl = wave >> 1, kh = wave & 1;
    const int src = rank ^ group_off(q, mem);
    const int rot = (mem == 0 && !foreign) ? 4 * cp : 0;
    if (!foreign) {
        const int wc = (lane & 3) ^ (((lane >> 5) & 1) << 1);
        const char* wl = pw + (size_t)(lane >> 2) * (NS * 2) + cp * 64 + wc * 16;
#pragma unroll
        for (int i = 0; i < 16; ++i) {
            const int blk = wave * 16 + i;
            glds16(wl + (size_t)((blk + 2 * rot) & 127) * 16 * (NS * 2), lds0 + LDS_W + blk * 1024);
        }
    }
    ORDER();
    const unsigned long long tab =
        *(const GAS unsigned long long*)((const char*)__builtin_amdgcn_kernarg_segment_ptr() + offsetof(Params, out) + (lane & 15) * 8);
    ORDER();
    __builtin_amdgcn_s_barrier();
    ORDER();
    const unsigned call = (unsigned)__builtin_amdgcn_readfirstlane(ticket) / NWG + 1;
    int own0 = rank, own1 = rank, slot0 = 0, slot1 = 0, col0 = cp * 32, col1 = cp * 32 + 16;
    if (!foreign) {
        asm volatile("s_waitcnt vmcnt(0)" ::: "memory");
        if (lane == 0) lds_add(sy + SY_WCNT, 1u);
    } else {
        int d0, d1;
        const int tf = (cp - 8) * 2;
        foreign_tile(q, tf, d0, slot0); foreign_tile(q, tf + 1, d1, slot1);
        own0 = rank ^ d0; own1 = rank ^ d1; col0 = tf * 16; col1 = tf * 16 + 16;
        lds_wait_ge(sy + SY_COMM, 4u);
        lds_wait_ge(sy + SY_WGO, 1u);
        const int t4 = wave >> 2;
        const char* wl = myws + WS_WST + (size_t)(call & 1) * WST_BYTES + (size_t)(t4 ? slot1 : slot0) * TILE_BYTES + lane * 16;
#pragma unroll
        for (int i = 0; i < 16; ++i) {
            const unsigned blk = 16 * (wave & 3) + i;
            glds16(wl + (size_t)blk * 1024, lds0 + LDS_W + t4 * TILE_BYTES + blk * 1024);
        }
        asm volatile("s_waitcnt vmcnt(0)" ::: "memory");
        if (lane == 0) lds_add(sy + SY_WCNT, 1u);
    }
    const bool t1 = (lane & 7) >= 4;
    char* op = pick(tab, t1 ? own1 : own0) + (size_t)(src * MS + q * 64 + rtl * 16 + kh * 8 + (lane >> 3)) * (NS * 2) + (t1 ? col1 : col0) * 2 + (lane & 3) * 8;
    char* fp = pick(tab, 8 + ((lane & 1) ? own1 : own0)) + WS_CTRL + CTRL_DFLAG + (size_t)(rank * 1024 + ((q * 4 + mem) * 8 + (cp & 7)) * 8 + wave) * 4;
    lds_wait_ge(sy + SY_XGO, 1u);
    const bool local = lds_ld(sy + SY_XGO) == 2;
    const bool bad = lds_ld(sy + SY_BAD) != 0;
    const char* abase = local ? px : (p.ws[src] + WS_XCOPY + (size_t)(call & 1) * SHARD_BYTES);
    const __amdgpu_buffer_rsrc_t ra = make_rsrc(abase, bad ? 0u : SHARD_BYTES);
    const unsigned voff = (q * 4 + rtl) * 65536 + (local ? ((lane & 15) * (KD * 2) + (lane >> 4) * 16) : lane * 16);
    const unsigned kstride = local ? 64u : 1024u;
    v4u a[32];
    v4f acc0 = {0.f, 0.f, 0.f, 0.f}, acc1 = {0.f, 0.f, 0.f, 0.f};
    auto issue = [&](int jb) {
#pragma unroll
        for (int i = 8 * jb; i < 8 * jb + 8; ++i)
            a[i] = __builtin_amdgcn_raw_buffer_load_b128(ra, voff, ((2 * i + kh + rot) & 63) * kstride, 0);
        ORDER();
    };
    const int g = lane >> 4, r4 = (lane >> 2) & 3, c4 = lane & 3;
    auto mma = [&](int i, const LAS char* p0l, const LAS char* p0h, const LAS char* p1l, const LAS char* p1h) {
        const v4s l0 = __builtin_amdgcn_ds_read_tr16_b64_v4i16((LAS v4s*)p0l);
        const v4s h0 = __builtin_amdgcn_ds_read_tr16_b64_v4i16((LAS v4s*)p0h);
        const v4s l1 = __builtin_amdgcn_ds_read_tr16_b64_v4i16((LAS v4s*)p1l);
        const v4s h1 = __builtin_amdgcn_ds_read_tr16_b64_v4i16((LAS v4s*)p1h);
        const v8s wf0 = __builtin_shufflevector(l0, h0, 0, 1, 2, 3, 4, 5, 6, 7);
        const v8s wf1 = __builtin_shufflevector(l1, h1, 0, 1, 2, 3, 4, 5, 6, 7);
        const v8bf xa = __builtin_bit_cast(v8bf, a[i]);
        acc0 = __builtin_amdgcn_mfma_f32_16x16x32_bf16(__builtin_bit_cast(v8bf, wf0), xa, acc0, 0, 0, 0);
        acc1 = __builtin_amdgcn_mfma_f32_16x16x32_bf16(__builtin_bit_cast(v8bf, wf1), xa, acc1, 0, 0, 0);
    };
    if (foreign) {
        lds_wait_ge(sy + SY_WCNT, 8u);
        if (wave == 0) STAMP(13);
#pragma unroll
        for (int jb = 0; jb < 4; ++jb) issue(jb);
        if (wave == 0) STAMP(6);
        const unsigned flo = LDS_W + kh * 1024 + (8 * g + r4) * 32 + c4 * 8;
        const LAS char* f0l = (const LAS char*)lds + flo + ((g & 1) * 128);
        const LAS char* f0h = (const LAS char*)lds + flo + (((g & 1) ^ 1) * 128);
        const LAS char* f1l = f0l + TILE_BYTES;
        const LAS char* f1h = f0h + TILE_BYTES;
#pragma unroll
        for (int i = 0; i < 32; ++i) mma(i, f0l + i * 2048, f0h + i * 2048, f1l + i * 2048, f1h + i * 2048);
    } else {
        const unsigned done = sy + SY_DONE + kh * 4;
#pragma unroll
        for (int jb = 0; jb < (WB < 4 ? WB : 4); ++jb) {
            const int need = 32 * rtl + 8 * jb + 8 - 8 * WB;
            if (need > 0) lds_wait_ge(done, (unsigned)need);
            issue(jb);
        }
        if (wave == 0) STAMP(6);
        lds_wait_ge(sy + SY_WCNT, 8u);
        const unsigned wlo = LDS_W + kh * 2048 + (8 * g + r4) * 64 + c4 * 8;
        const LAS char* w0p = (const LAS char*)lds + wlo + ((g & 1) * 32);
        const LAS char* w1p = (const LAS char*)lds + wlo + (((g & 1) ^ 1) * 32);
#pragma unroll
        for (int jb = 0; jb < 4; ++jb) {
#pragma unroll
            for (int i = 8 * jb; i < 8 * jb + 8; ++i) mma(i, w0p + i * 4096, w0p + i * 4096 + 256, w1p + i * 4096, w1p + i * 4096 + 256);
            ORDER();
            if (lane == 0) lds_max(done, 32 * rtl + 8 * jb + 8);
            if (jb + WB < 4) issue(jb + WB);
        }
    }
    if (wave == 0) STAMP(8);
    if (wave == 7) STAMP(9);
    {
        char* red = lds + LDS_RED + rtl * (2 * 16 * RED_ROW);
        char* mine = red + kh * (16 * RED_ROW) + (lane & 15) * RED_ROW + (lane >> 4) * 16;
        *(v4f*)mine = acc0;
        *(v4f*)(mine + 64) = acc1;
        if (lane == 0) lds_st(sy + SY_PAIR + wave * 4, 1u);
        lds_wait_ge(sy + SY_PAIR + (wave ^ 1) * 4, 1u);
        const int row = kh * 8 + (lane >> 3);
        const v4f s0 = *(const v4f*)(red + row * RED_ROW + (lane & 7) * 16);
        const v4f s1 = *(const v4f*)(red + 16 * RED_ROW + row * RED_ROW + (lane & 7) * 16);
        const v4f y = s0 + s1;
        v2u o;
        if (!bad) { o.x = pack_bf16(silu(y[0]), silu(y[1])); o.y = pack_bf16(silu(y[2]), silu(y[3])); }
        else { o.x = o.y = 0x7FC07FC0u; }
        if (!foreign) {
            *(GAS v2u*)op = o;
        } else {
            asm volatile("global_store_dwordx2 %0, %1, off sc0 sc1\n\ts_waitcnt vmcnt(0)" : : "v"(op), "v"(o) : "memory");
            if (lane < 2 && (lane == 0 || own1 != own0)) st_sys((gu32*)fp, call);
        }
    }
    if (wave == 7) STAMP(10);
}

}

extern "C" void kernel_launch(int rank, int nranks, ncclComm_t comm,
                              hipStream_t stream,
                              void* const* d_in, const int* in_sizes, int n_in,
                              void* d_out, int out_size,
                              const int* in_dtypes, int out_dtype,
                              void* const* d_out_all,
                              void* const* d_ws_all, size_t ws_bytes) {
    if (nranks != NR || n_in != 2 || ws_bytes < WS_NEED || in_sizes[0] != MS * KD || in_sizes[1] != KD * NS ||
        out_size != NR * MS * NS || (unsigned)rank >= (unsigned)NR) {
        fprintf(stderr, "kernel_launch: unexpected problem geometry\n");
        return;
    }
    HIP_CHECK(hipFuncSetAttribute((const void*)ag_gemm_silu, hipFuncAttributeMaxDynamicSharedMemorySize, LDS_BYTES));
    Params p;
    p.x = (const char*)d_in[0];
    p.w = (const char*)d_in[1];
    for (int i = 0; i < NR; ++i) { p.out[i] = (char*)d_out_all[i]; p.ws[i] = (char*)d_ws_all[i]; }
    p.out[rank] = (char*)d_out;
    p.myws = (char*)d_ws_all[rank];
    p.rank = rank;
    hipLaunchKernelGGL(ag_gemm_silu, dim3(NWG), dim3(NTHR), LDS_BYTES, stream, p);
    HIP_CHECK(hipGetLastError());
}
```
